```python
import numpy as np

kernel_name = "gpu_gemm_ar_m4096_k4096_n16384_e4m3e5m2_dequant"


ABSMAX_RTOL = 0.05

COMPUTE_DTYPE = "float32"
EPILOGUE = "dequant"
IN_QUANT = None
OUT_QUANT = None
IN_FP8 = "e4m3e5m2"
IN_DTYPES = {"x": "e4m3", "w_mat": "e5m2",
             "scale_x": "float32", "scale_w": "float32"}
WORLD_SIZE = 8

def _snap_fp8(x, mant_bits, max_abs, min_normal):
    a = np.abs(x.astype(np.float64))
    _m, _e = np.frexp(a)
    q = np.where(a >= min_normal, np.ldexp(1.0, _e - 1 - mant_bits),
                 min_normal * 2.0 ** -mant_bits)
    snapped = np.minimum(np.round(a / q) * q, max_abs)
    return (np.sign(x) * snapped).astype(np.float32)

def setup_inputs():

    x = _snap_fp8(np.random.randn(4096, 4096).astype(np.float32), 3, 448.0, 2.0 ** -6)
    w_mat = _snap_fp8(np.random.randn(4096, 16384).astype(np.float32), 2, 57344.0, 2.0 ** -14)
    scale_x = (0.02 * (1.0 + np.random.rand(1))).astype(np.float32)
    scale_w = (0.02 * (1.0 + np.random.rand(1))).astype(np.float32)
    return {"x": x, "w_mat": w_mat, "scale_x": scale_x, "scale_w": scale_w}

def reference(x, w_mat, scale_x, scale_w):
    acc = (x @ w_mat).astype(np.float32)
    y = (acc * (scale_x * scale_w)).astype(np.float32)
    return y
```

```cpp
#include <hip/hip_runtime.h>
#include <rccl/rccl.h>
#include <cstdio>
#include <cstdint>

#define HIP_CHECK(expr) do { hipError_t _e = (expr); if (_e != hipSuccess) { \
    fprintf(stderr, "HIP error %s at %s:%d\n", hipGetErrorString(_e), __FILE__, __LINE__); \
    return; } } while (0)

namespace gk {
#define LAS __attribute__((address_space(3)))
#define GAS __attribute__((address_space(1)))
typedef int i32x4 __attribute__((ext_vector_type(4)));
typedef int i32x8 __attribute__((ext_vector_type(8)));
typedef unsigned u32x4 __attribute__((ext_vector_type(4)));
typedef float f32x4 __attribute__((ext_vector_type(4)));

#define MON_SLEEP 1
#define POLL_SLEEP 2
#define SKEW 4
#define OUT_AUX 2
#define LD_AUX 16
#define RSKEW 9216
constexpr int GM = 4096, GN = 16384, KS = 512, NRK = 8;
constexpr unsigned A_BYTES = 16u * 32u * 4096u;
constexpr unsigned REGION = A_BYTES + 256u * 32u * 1024u;
constexpr unsigned RSTRIDE = REGION + RSKEW;
constexpr int GRID = 256, NCOMM = 16, NPUSH = 15, NPW = NPUSH * 4;
constexpr int NUNIT = REGION / 1024;
constexpr int NPAIR = NUNIT / 2;
constexpr int NG = 160;
constexpr int ST1 = 32 - 3 * SKEW, ST2 = 32 - 2 * SKEW, ST3 = 32 - SKEW;
static_assert(SKEW >= 4 && SKEW % 2 == 0 && SKEW <= 8, "a parked strip drains in 4 K-steps");
constexpr size_t WS_CTR = 0;
constexpr size_t WS_TMO = 16384;
constexpr size_t WS_CLAIM = 16384 + 256;
constexpr size_t WS_FLAG = 32768;
constexpr size_t WS_PROG = 65536;
constexpr size_t WS_DATA = 1u << 20;
constexpr size_t WS_NEED = WS_DATA + 2ull * NRK * RSTRIDE;
constexpr unsigned long long DEADLINE = 100000000ull;
constexpr int LDS_BYTES = 131072;

struct Params {
    const unsigned char* x; const unsigned char* w; const float* sx; const float* sw; float* out;
    char* ws[NRK];
    int rank;
};

__host__ __device__ __forceinline__ constexpr int units_of_step(int g) {
    return (g < 32 ? 64 : 0) + 16 + ((g >= ST1 && g < ST1 + 128) ? 15 : 0) + ((g >= ST2 && g < ST2 + 128) ? 15 : 0) + ((g >= ST3 && g < ST3 + 128) ? 14 : 0);
}
constexpr int SEG_G[9] = {0, ST1, ST2, ST3, 32, ST1 + 128, ST2 + 128, ST3 + 128, NG};
__host__ __device__ constexpr int seg_base(int s) { int b = 0; for (int i = 0; i < s; ++i) b += (SEG_G[i + 1] - SEG_G[i]) * units_of_step(SEG_G[i]); return b; }
static_assert(seg_base(8) == NUNIT, "unit count");
__device__ __forceinline__ void unit_of(int n, unsigned& dst, unsigned& src) {
    int base = 0, g0 = 0, per = units_of_step(0);
#pragma unroll
    for (int s = 1; s < 8; ++s) if (n >= seg_base(s)) { base = seg_base(s); g0 = SEG_G[s]; per = units_of_step(SEG_G[s]); }
    const int r = n - base;
    const int q = (int)(((float)r + 0.5f) * (1.0f / (float)per));
    const int g = g0 + q, t = g & 31;
    int idx = r - q * per;
    const bool isx = g < 32 && idx < 64;
    const int j = g < 32 ? idx - 64 : idx;
    const int a1 = (g >= ST1 && g < ST1 + 128) ? 15 : 0, a2 = (g >= ST2 && g < ST2 + 128) ? 15 : 0;
    const int strip = j < 15 ? 15 * (g >> 5) + j : j == 15 ? 75 + (g >> 5) : j < 16 + a1 ? 80 + 15 * ((g - ST1) >> 5) + j - 16
                    : j < 16 + a1 + a2 ? 140 + 15 * ((g - ST2) >> 5) + j - 16 - a1 : 200 + 14 * ((g - ST3) >> 5) + j - 16 - a1 - a2;
    dst = isx ? (unsigned)t * 65536u + (unsigned)idx * 1024u : A_BYTES + (unsigned)(n - 64 * min(g + 1, 32)) * 1024u;
    src = isx ? (unsigned)idx * (64u * KS) + (unsigned)t * 16u : (unsigned)(t * 16) * GN + (unsigned)strip * 64u;
}

__device__ __forceinline__ __amdgpu_buffer_rsrc_t mk_rsrc(const void* p, unsigned bytes) {
    return __builtin_amdgcn_make_buffer_rsrc((void*)p, 0, bytes, 0x00020000);
}
constexpr int AUX_SYS = 17;

__device__ __forceinline__ void pusher(const Params& p, char* myws, volatile LAS unsigned* oldest, unsigned call, int wid, int lane) {
    const int gw = (blockIdx.x - 1) * 4 + wid;
    const size_t dbase = WS_DATA + (size_t)((call & 1u) * NRK + p.rank) * RSTRIDE;
    __amdgpu_buffer_rsrc_t rd[NRK];
#pragma unroll
    for (int q = 0; q < NRK; ++q) rd[q] = mk_rsrc(p.ws[(p.rank + 1 + q + gw) & 7] + dbase, REGION);
    char* wsl = p.ws[0];
#pragma unroll
    for (int q = 1; q < NRK; ++q) wsl = (lane & 7) == q ? p.ws[q] : wsl;
    GAS unsigned* flag = (GAS unsigned*)(wsl + WS_FLAG) + p.rank * 16 + blockIdx.x;
    GAS unsigned* claim = (GAS unsigned*)(myws + WS_CLAIM);
    const unsigned cbase = (call - 1u) * (unsigned)NPAIR - 2u * NPW;
    const int g = lane >> 4, c16 = lane & 15;
    const unsigned voffA = (unsigned)(4 * c16) * KS + (unsigned)(4 * g);
    const unsigned voffB = (unsigned)(4 * g) * GN + (unsigned)(4 * c16);
    const unsigned vstA = (unsigned)(lane * 16);
    const unsigned vstB = (unsigned)((16 * g + (c16 ^ (4 * g))) * 16);
    u32x4 ld[2]; unsigned dst[2], vD, vS;
#define P_DECODE(pair) unit_of(min(2 * (pair) + (lane & 1), NUNIT - 1), vD, vS)
#define P_LOAD(i) do { dst[i] = __builtin_amdgcn_readlane(vD, i); const unsigned _src = __builtin_amdgcn_readlane(vS, i); \
        const bool _x = dst[i] < A_BYTES; const auto _r = mk_rsrc(_x ? p.x : p.w, _x ? GM * KS : KS * GN); const unsigned _v = _x ? voffA : voffB; \
        ld[i].x = __builtin_amdgcn_raw_buffer_load_b32(_r, _v, _src, 0); ld[i].y = __builtin_amdgcn_raw_buffer_load_b32(_r, _v, _src + (_x ? 2 * KS : GN), 0); \
        ld[i].z = __builtin_amdgcn_raw_buffer_load_b32(_r, _v, _src + (_x ? KS : 2 * GN), 0); ld[i].w = __builtin_amdgcn_raw_buffer_load_b32(_r, _v, _src + (_x ? 3 * KS : 3 * GN), 0); } while (0)
#define P_SEND(i) do { \
        const bool _x = dst[i] < A_BYTES; \
        const unsigned s1 = _x ? 0x03020100u : 0x05010400u, s2 = _x ? 0x07060504u : 0x07030602u, s3 = _x ? 0x03020100u : 0x05040100u, s4 = _x ? 0x07060504u : 0x07060302u; \
        const unsigned t0 = __builtin_amdgcn_perm(ld[i].y, ld[i].x, s1), t1 = __builtin_amdgcn_perm(ld[i].y, ld[i].x, s2); \
        const unsigned t2 = __builtin_amdgcn_perm(ld[i].w, ld[i].z, s1), t3 = __builtin_amdgcn_perm(ld[i].w, ld[i].z, s2); \
        unsigned d0 = __builtin_amdgcn_perm(t2, t0, s3), d1 = __builtin_amdgcn_perm(t2, t0, s4); \
        unsigned d2 = __builtin_amdgcn_perm(t3, t1, s3), d3 = __builtin_amdgcn_perm(t3, t1, s4); \
        auto r0 = __builtin_amdgcn_permlane32_swap(d0, d2, false, false); d0 = r0[0]; d2 = r0[1]; \
        auto r1 = __builtin_amdgcn_permlane32_swap(d1, d3, false, false); d1 = r1[0]; d3 = r1[1]; \
        auto r2 = __builtin_amdgcn_permlane16_swap(d0, d1, false, false); d0 = r2[0]; d1 = r2[1]; \
        auto r3 = __builtin_amdgcn_permlane16_swap(d2, d3, false, false); d2 = r3[0]; d3 = r3[1]; \
        u32x4 v = (u32x4){d0, d1, d2, d3}; unsigned vst = (_x ? vstA : vstB) + dst[i]; \
        asm volatile("" : "+v"(v), "+v"(vst) :: "memory"); \
        P_LOAD(i); \
        asm volatile("" ::: "memory"); \
          \
        _Pragma("unroll") for (int q = 0; q < NRK; ++q) __builtin_amdgcn_raw_buffer_store_b128(v, rd[q], vst, 0, AUX_SYS); \
        asm volatile("" ::: "memory"); } while (0)
#define P_PUBLISH(u) do { if (lane == 0) oldest[wid] = (unsigned)(u); \
        if (wid == 0) { const unsigned _a = oldest[1], _b = oldest[2], _c = oldest[3]; const unsigned _m = min(min((unsigned)(u), _a), min(_b, _c)); \
            if (lane < NRK) __hip_atomic_store(flag, (call << 14) + _m, __ATOMIC_RELAXED, __HIP_MEMORY_SCOPE_SYSTEM); } } while (0)
    int prev = 0, cur = gw;
    unsigned ticket = cbase + NPW + gw;
    P_DECODE(cur); P_LOAD(0); P_LOAD(1);
    bool first = true;
    for (;;) {
        asm volatile("s_waitcnt vmcnt(20)" : "+v"(ticket) :: "memory");
        const int nxt = (int)min((unsigned)__builtin_amdgcn_readfirstlane(ticket) - cbase, (unsigned)NPAIR);
        if (!first) P_PUBLISH(2 * prev);
        {
            unsigned long long ex;
            asm volatile("s_mov_b64 %1, exec\n\ts_mov_b64 exec, 1\n\tv_mov_b32 %0, 1\n\tglobal_atomic_add %0, %2, %0, %3 sc0\n\ts_mov_b64 exec, %1"
                         : "=&v"(ticket), "=&s"(ex) : "v"(0u), "s"(claim) : "memory");
        }
        P_DECODE(nxt);
        P_SEND(0);
        asm volatile("s_waitcnt vmcnt(20)" ::: "memory");
        if (!first) P_PUBLISH(2 * prev + 1);
        P_SEND(1);
        first = false;
        prev = cur; cur = nxt;
        if (cur >= NPAIR) break;
    }
    asm volatile("s_waitcnt vmcnt(8)" ::: "memory");
    P_PUBLISH(2 * prev + 1);
    asm volatile("s_waitcnt vmcnt(0)" ::: "memory");
    if (lane == 0) oldest[wid] = (unsigned)NUNIT;
    if (wid == 0) {
        const unsigned long long t0 = __builtin_amdgcn_s_memrealtime();
        unsigned last = 0;
        for (;;) {
            const unsigned m = __builtin_amdgcn_readfirstlane(min(oldest[1], min(oldest[2], oldest[3])));
            if (m != last && lane < NRK) __hip_atomic_store(flag, (call << 14) + m, __ATOMIC_RELAXED, __HIP_MEMORY_SCOPE_SYSTEM);
            last = m;
            if (m >= (unsigned)NUNIT || __builtin_amdgcn_s_memrealtime() - t0 > DEADLINE) break;
            __builtin_amdgcn_s_sleep(1);
        }
    }
#undef P_DECODE
#undef P_LOAD
#undef P_SEND
#undef P_PUBLISH
}

__device__ __forceinline__ void monitor(const Params& p, char* ws, unsigned call, int wid, int lane) {
    GAS const unsigned long long* fl = (GAS const unsigned long long*)(ws + WS_FLAG) + lane;
    GAS unsigned* tmo = (GAS unsigned*)(ws + WS_TMO);
    GAS unsigned* prog = (GAS unsigned*)(ws + WS_PROG + (size_t)(NCOMM + wid * 60 + (lane < 60 ? lane : 0)) * 128);
    const unsigned fbase = call << 14;
    const unsigned long long t0 = __builtin_amdgcn_s_memrealtime();
    int last = 0; unsigned polls = 0;
    for (;;) {
        const unsigned long long f = __hip_atomic_load(fl, __ATOMIC_RELAXED, __HIP_MEMORY_SCOPE_SYSTEM);
        int m = (int)((unsigned)(f >> 32) - fbase);
        if (lane & 7) m = min(m, (int)((unsigned)f - fbase));
#pragma unroll
        for (int o = 32; o >= 1; o >>= 1) m = min(m, __shfl_xor(m, o));
        const int avail = min(__builtin_amdgcn_readfirstlane(m), NUNIT);
        if (avail > last) {
            last = avail;
            __builtin_amdgcn_fence(__ATOMIC_ACQUIRE, "");
            if (lane < 60) __hip_atomic_store(prog, fbase + (unsigned)avail, __ATOMIC_RELAXED, __HIP_MEMORY_SCOPE_AGENT);
            if (avail >= NUNIT) break;
        } else if ((++polls & 63u) == 0u) {
            if ((unsigned)__builtin_amdgcn_readfirstlane(__hip_atomic_load(tmo, __ATOMIC_RELAXED, __HIP_MEMORY_SCOPE_AGENT)) == call
                || __builtin_amdgcn_s_memrealtime() - t0 > DEADLINE) {
                if (lane == 0) __hip_atomic_store(tmo, call, __ATOMIC_RELAXED, __HIP_MEMORY_SCOPE_AGENT);
                break;
            }
        }
        __builtin_amdgcn_s_sleep(MON_SLEEP);
    }
}

__device__ __noinline__ int wait_units(char* ws, unsigned call, int need) {
    GAS const unsigned* prog = (GAS const unsigned*)(ws + WS_PROG + (size_t)blockIdx.x * 128);
    GAS unsigned* tmo = (GAS unsigned*)(ws + WS_TMO);
    const unsigned long long t0 = __builtin_amdgcn_s_memrealtime();
    int avail; unsigned polls = 0;
    for (;;) {
        avail = (int)((unsigned)__builtin_amdgcn_readfirstlane(__hip_atomic_load(prog, __ATOMIC_RELAXED, __HIP_MEMORY_SCOPE_AGENT)) - (call << 14));
        if (avail >= need) break;
        if ((++polls & 63u) == 0u) {
            if ((unsigned)__builtin_amdgcn_readfirstlane(__hip_atomic_load(tmo, __ATOMIC_RELAXED, __HIP_MEMORY_SCOPE_AGENT)) == call
                || __builtin_amdgcn_s_memrealtime() - t0 > DEADLINE) {
                __hip_atomic_store(tmo, call, __ATOMIC_RELAXED, __HIP_MEMORY_SCOPE_AGENT);
                avail = -1; break;
            }
        }
        __builtin_amdgcn_s_sleep(POLL_SLEEP);
    }
    __builtin_amdgcn_fence(__ATOMIC_ACQUIRE, "wavefront");
    return avail;
}

__global__ __launch_bounds__(512, 2) void fused_kernel(Params p) {
    extern __shared__ __attribute__((aligned(16))) unsigned char shm[];
    LAS unsigned char* lds = (LAS unsigned char*)shm;
    const int tid = threadIdx.x, wid = __builtin_amdgcn_readfirstlane(tid >> 6), lane = tid & 63;
    char* myws = p.ws[0];
#pragma unroll
    for (int q = 1; q < NRK; ++q) myws = p.rank == q ? p.ws[q] : myws;
    unsigned call = 0;
    if (lane == 0) call = __hip_atomic_fetch_add((GAS unsigned*)(myws + WS_CTR) + blockIdx.x * 8 + wid, 1u, __ATOMIC_RELAXED, __HIP_MEMORY_SCOPE_AGENT) + 1u;
    call = __builtin_amdgcn_readfirstlane(call);
    constexpr bool do_push = true, do_gemm = true, do_mon = true;
    if (blockIdx.x < NCOMM) {
        if (blockIdx.x == 0) { if (wid < 4 && do_mon) monitor(p, myws, call, wid, lane); return; }
        volatile LAS unsigned* oldest = (volatile LAS unsigned*)lds;
        if (tid < 4) oldest[tid] = 0u;
        __syncthreads();
        if (wid < 4 && do_push) pusher(p, myws, oldest, call, wid, lane);
        return;
    }
    if (!do_gemm) return;
    const int lin = 30 * (blockIdx.x & 7) + (blockIdx.x >> 3) - NCOMM / 8;
    const int pm = 4 * (lin / 60) + (lin & 3), wcol = (lin % 60) >> 2;
    const int wr = wid & 3, wc = wid >> 2, fr = lane & 15, fq = lane >> 4;
    float scale = p.sx[0] * p.sw[0];
    const auto rsrc = mk_rsrc(myws + WS_DATA + (size_t)((call & 1u) * NRK + wid) * RSTRIDE, REGION);
    const auto rout = mk_rsrc(p.out, (unsigned)GM * GN * 4u);
    const unsigned vld = lane * 16;
    const int aoff = fq * 4096 + (wr * 64 + fr) * 16;
    const int boff0 = 65536 + fq * 1024 + (fr & 3) * 256 + (((wc * 2) ^ (fr & 3)) * 4 + (fr >> 2)) * 16;
    const int boff1 = boff0 ^ 64;
    const unsigned vout = (unsigned)(pm * 256 + wr * 64 + 4 * fr) * (GN * 4u) + (unsigned)(wc * 32 + 4 * fq) * 4u;
    const int st3 = wcol == 0 ? 0 : ST3, en3 = wcol == 0 ? NG : ST3 + 128;
    const int s30 = wcol == 0 ? 75 : 199 + wcol, s3d = wcol == 0 ? 1 : 14;
    const unsigned aBase = (unsigned)pm * 4096u;
    int avail = 0;
    unsigned tabC[2], tabP[2], tabQ[2], tabN[2];
    unsigned p0, q0, n0;
    {
        auto strip_of = [&](int q, int g) { return q == 0 ? 15 * (g >> 5) + wcol : q == 1 ? 80 + 15 * ((g - ST1) >> 5) + wcol : q == 2 ? 140 + 15 * ((g - ST2) >> 5) + wcol : s30 + s3d * ((g - st3) >> 5); };
        auto act = [&](int q, int g) { return q == 0 ? (g >= 0 && g < NG) : q == 1 ? (unsigned)(g - ST1) < 128u : q == 2 ? (unsigned)(g - ST2) < 128u : (g >= st3 && g < en3); };
        auto ends = [&](int q, int g) { return act(q, g) && ((g - (q == 0 ? 0 : q == 1 ? ST1 : q == 2 ? ST2 : st3)) & 31) == 30; };
        auto parks = [&](int g) { return ends(0, g - 2) || ends(1, g - 2) || ends(2, g - 2) || ends(3, g - 2); };
        auto cumw = [&](int g) { return 16 * g + 15 * min(max(g - ST1, 0), 128) + 15 * min(max(g - ST2, 0), 128) + 14 * min(max(g - ST3, 0), 128); };
        auto words = [&](int g, unsigned& P, unsigned& Q, unsigned& N) {
            const int c = cumw(g), i0 = c + wcol, a1 = act(1, g) ? 15 : 0, a2 = act(2, g) ? 15 : 0;
            const int i1 = a1 ? c + 16 + wcol : i0, i2 = a2 ? c + 16 + a1 + wcol : i0, i3 = !act(3, g) ? i0 : wcol == 0 ? c + 15 : c + 15 + a1 + a2 + wcol;
            const int dw = cumw(g + 1) - c, dn = dw + (g < 32 ? 64 : 0);
            P = (unsigned)i0 | ((unsigned)i1 << 16); Q = (unsigned)i2 | ((unsigned)i3 << 16);
            N = (unsigned)(cumw(g + 1) + 64 * min(g + 1, 32)) | ((unsigned)dn << 14) | ((unsigned)dw << 21); };
#pragma unroll
        for (int h = 0; h < 2; ++h) {
            const int g = 2 * (lane + 64 * h);
            const bool f0 = ends(0, g - 2), f1 = ends(1, g - 2), f2 = ends(2, g - 2), f3 = ends(3, g - 2);
            unsigned c = (act(1, g) ? 1u : 0u) | (act(2, g) ? 2u : 0u) | (act(3, g) ? 4u : 0u) | (f0 ? 8u : 0u) | (f1 ? 16u : 0u) | (f2 ? 32u : 0u);
            c |= (f3 ? (f0 ? 128u : 64u) : 0u) | (parks(g - 2) ? 256u : 0u) | ((parks(g - 4) && !parks(g)) ? 512u : 0u);
            c |= (unsigned)(strip_of(f0 ? 0 : f1 ? 1 : f2 ? 2 : 3, g - 2) & 255) << 16;
            c |= (unsigned)(strip_of(3, g - 2) & 255) << 24;
            tabC[h] = c;
            words(min(g + 2, NG - 2), tabP[h], tabQ[h], tabN[h]);
        }
        words(0, p0, q0, n0);
        p0 = __builtin_amdgcn_readfirstlane(p0); q0 = __builtin_amdgcn_readfirstlane(q0); n0 = __builtin_amdgcn_readfirstlane(n0);
    }
#define ULO(w) (A_BYTES + (((w) & 0xffffu) << 10))
#define UHI(w) (A_BYTES + (((w) >> 16) << 10))
#define N_FIRST(n) ((int)((n) & 0x3fffu))
#define N_SECOND(n) ((int)(((n) & 0x3fffu) + (((n) >> 14) & 127u)))
#define N_DW(n) (((n) >> 21) << 10)
#define GLDS(ldsoff, soff, imm) __builtin_amdgcn_raw_ptr_buffer_load_lds(rsrc, (LAS void*)(lds + (ldsoff)), 16, vld, (soff), (imm), LD_AUX)
#define STA(b, soff) do { GLDS((b) * 32768 + wid * 4096, soff, 0); GLDS((b) * 32768 + wid * 4096, soff, 1024); GLDS((b) * 32768 + wid * 4096, soff, 2048); GLDS((b) * 32768 + wid * 4096, soff, 3072); } while (0)
#define STB(b, q, soff) GLDS(65536 + ((b) * 4 + (q)) * 8192 + wid * 1024, soff, 0)
#define LD8(ptr, d) __builtin_shufflevector(*(const LAS i32x4*)(ptr), *(const LAS i32x4*)((ptr) + (d)), 0, 1, 2, 3, 4, 5, 6, 7)
#define LDA(b) do { _Pragma("unroll") for (int m = 0; m < 4; ++m) At[m] = LD8(lds + (b) * 32768 + aoff + m * 256, 16384); } while (0)
#define LDB(b, q) do { Bt[(q) & 1][0] = LD8(lds + ((b) * 4 + (q)) * 8192 + boff0, 4096); Bt[(q) & 1][1] = LD8(lds + ((b) * 4 + (q)) * 8192 + boff1, 4096); } while (0)
#define MMA(q) do { _Pragma("unroll") for (int m = 0; m < 4; ++m) _Pragma("unroll") for (int n = 0; n < 2; ++n) \
        acc[q][m][n] = __builtin_amdgcn_mfma_scale_f32_16x16x128_f8f6f4(Bt[(q) & 1][n], At[m], acc[q][m][n], 1, 0, 0, 0, 0, 0); } while (0)
#define STO(v, voff) __builtin_amdgcn_raw_buffer_store_b128(__builtin_bit_cast(u32x4, (v)), rout, (voff), 0, OUT_AUX)
#define FLUSH(q, strip) do { const int _vo = (int)(vout + (unsigned)(strip) * 256u); _Pragma("unroll") for (int m = 0; m < 4; ++m) { const int _vr = _vo + m * GN * 4; \
        _Pragma("unroll") for (int n = 0; n < 2; ++n) { STO(acc[q][m][n] * scale, _vr + n * 64); acc[q][m][n] = (f32x4){0.f, 0.f, 0.f, 0.f}; } } } while (0)
#define PARK(q, strip) do { vpk = (int)(vout + (unsigned)(strip) * 256u); _Pragma("unroll") for (int m = 0; m < 4; ++m) \
        _Pragma("unroll") for (int n = 0; n < 2; ++n) { pk[m][n] = acc[q][m][n] * scale; acc[q][m][n] = (f32x4){0.f, 0.f, 0.f, 0.f}; } } while (0)
#define DRIP(k) do { SCHED; STO(pk[(k) >> 1][(k) & 1], vpk + ((k) >> 1) * GN * 4 + ((k) & 1) * 64); SCHED; } while (0)
#define WAIT_V(n) asm volatile("s_waitcnt vmcnt(" #n ")" ::: "memory")
#define WAIT_L0 __builtin_amdgcn_s_waitcnt(0xC07F)
#define BAR __builtin_amdgcn_s_barrier()
#define SCHED __builtin_amdgcn_sched_barrier(0)
#define PRIO(n) __builtin_amdgcn_s_setprio(n)
#define NEED(n) do { const int _n = (n); if (_n > avail) { avail = __builtin_amdgcn_readfirstlane(wait_units(myws, call, _n)); \
        if (avail < 0) { avail = 0x7fffffff; scale = __builtin_nanf(""); } } } while (0)
#define STAMP() do {} while (0)
    STAMP();
    constexpr int VPK_OFF = 0x40000000;
    f32x4 acc[4][4][2], pk[4][2]; int vpk = VPK_OFF;
#pragma unroll
    for (int m = 0; m < 4; ++m)
#pragma unroll
        for (int n = 0; n < 2; ++n) { pk[m][n] = (f32x4){0.f, 0.f, 0.f, 0.f};
#pragma unroll
            for (int q = 0; q < 4; ++q) acc[q][m][n] = (f32x4){0.f, 0.f, 0.f, 0.f}; }
    i32x8 At[4], Bt[2][2];
    NEED(N_FIRST(n0));
    STA(0, aBase); STB(0, 0, ULO(p0)); STB(0, 1, UHI(p0)); DRIP(0); STB(0, 2, ULO(q0)); STB(0, 3, UHI(q0)); DRIP(0);
    NEED(N_SECOND(n0));
    STA(1, aBase + 65536); STB(1, 0, ULO(p0) + N_DW(n0)); STB(1, 1, UHI(p0) + N_DW(n0)); DRIP(0);
    unsigned c2 = ULO(q0) + N_DW(n0), c3 = UHI(q0) + N_DW(n0);
    WAIT_V(11); BAR;
    if (wc == 1) BAR;
    STAMP();
    for (int it = 0; it < NG / 2; ++it) {
        if (it == 64) { tabC[0] = tabC[1]; tabP[0] = tabP[1]; tabQ[0] = tabQ[1]; tabN[0] = tabN[1]; asm volatile("" : "+v"(tabC[0]), "+v"(tabP[0]), "+v"(tabQ[0]), "+v"(tabN[0])); }
        const unsigned ctl = __builtin_amdgcn_readlane(tabC[0], it & 63);
        const unsigned pw = __builtin_amdgcn_readlane(tabP[0], it & 63), qw = __builtin_amdgcn_readlane(tabQ[0], it & 63), nd = __builtin_amdgcn_readlane(tabN[0], it & 63);
        const unsigned na = aBase + (((unsigned)(2 * it + 2) & 31u) << 16), dw = N_DW(nd);
        LDB(0, 0); LDA(0); LDB(0, 1); STB(1, 2, c2); STB(1, 3, c3);
        if (ctl & 0x3f8u) {
            if (ctl & 8u) PARK(0, (ctl >> 16) & 255u);
            if (ctl & 16u) PARK(1, (ctl >> 16) & 255u);
            if (ctl & 32u) PARK(2, (ctl >> 16) & 255u);
            if (ctl & 64u) PARK(3, (ctl >> 16) & 255u);
            if (ctl & 128u) FLUSH(3, ctl >> 24);
            if (ctl & 256u) { pk[0][0] = pk[2][0]; pk[0][1] = pk[2][1]; pk[1][0] = pk[3][0]; pk[1][1] = pk[3][1]; vpk += 2 * GN * 4; }
            if (ctl & 512u) vpk = VPK_OFF;
        }
        DRIP(0);
        WAIT_V(11); WAIT_L0; BAR; PRIO(1); MMA(0); if (ctl & 1u) MMA(1); PRIO(0); BAR; SCHED;
        NEED(N_FIRST(nd));
        LDB(0, 2); LDB(0, 3); STA(0, na); STB(0, 0, ULO(pw)); STB(0, 1, UHI(pw)); DRIP(1);
        WAIT_V(11); WAIT_L0; BAR; PRIO(1); if (ctl & 2u) MMA(2); if (ctl & 4u) MMA(3); PRIO(0); BAR; SCHED;
        c2 = ULO(qw); c3 = UHI(qw);
        LDB(1, 0); LDA(1); LDB(1, 1); STB(0, 2, c2); STB(0, 3, c3); DRIP(2);
        WAIT_V(11); WAIT_L0; BAR; PRIO(1); MMA(0); if (ctl & 1u) MMA(1); PRIO(0); BAR; SCHED;
        NEED(N_SECOND(nd));
        LDB(1, 2); LDB(1, 3); STA(1, na + 65536); STB(1, 0, ULO(pw) + dw); STB(1, 1, UHI(pw) + dw); DRIP(3);
        WAIT_V(11); WAIT_L0; BAR; PRIO(1); if (ctl & 2u) MMA(2); if (ctl & 4u) MMA(3); PRIO(0); BAR; SCHED;
        c2 += dw; c3 += dw;
    }
    FLUSH(0, 60 + wcol);
    if (wcol == 0) FLUSH(3, 79);
    WAIT_V(0);
    STAMP();
    if (wc == 0) BAR;
}
}

extern "C" void kernel_launch(int rank, int nranks, ncclComm_t comm,
                              hipStream_t stream,
                              void* const* d_in, const int* in_sizes, int n_in,
                              void* d_out, int out_size,
                              const int* in_dtypes, int out_dtype,
                              void* const* d_out_all,
                              void* const* d_ws_all, size_t ws_bytes) {
    using namespace gk;
    if (nranks != NRK || (unsigned)rank >= (unsigned)NRK || n_in != 4 || ws_bytes < WS_NEED ||
        in_sizes[0] != GM * KS || in_sizes[1] != KS * GN || out_size != GM * GN) {
        fprintf(stderr, "kernel_launch: unexpected problem shape\n"); return;
    }
    static bool attr_set = false;
    if (!attr_set) {
        HIP_CHECK(hipFuncSetAttribute((const void*)fused_kernel, hipFuncAttributeMaxDynamicSharedMemorySize, LDS_BYTES));
        attr_set = true;
    }
    Params p{};
    p.x = (const unsigned char*)d_in[0]; p.w = (const unsigned char*)d_in[1];
    p.sx = (const float*)d_in[2]; p.sw = (const float*)d_in[3];
    p.out = (float*)d_out; p.rank = rank;
    for (int q = 0; q < NRK; ++q) p.ws[q] = (char*)d_ws_all[q];
    hipLaunchKernelGGL(fused_kernel, dim3(GRID), dim3(512), LDS_BYTES, stream, p);
    HIP_CHECK(hipGetLastError());
}
```

```cpp
#include <hip/hip_runtime.h>
#include <rccl/rccl.h>
#include <cstdio>
#include <cstdint>

#define HIP_CHECK(expr) do { hipError_t _e = (expr); if (_e != hipSuccess) { \
    fprintf(stderr, "HIP error %s at %s:%d\n", hipGetErrorString(_e), __FILE__, __LINE__); \
    return; } } while (0)

namespace gk {
#define LAS __attribute__((address_space(3)))
#define GAS __attribute__((address_space(1)))
typedef int i32x4 __attribute__((ext_vector_type(4)));
typedef int i32x8 __attribute__((ext_vector_type(8)));
typedef unsigned u32x4 __attribute__((ext_vector_type(4)));
typedef float f32x4 __attribute__((ext_vector_type(4)));
typedef float f32x2 __attribute__((ext_vector_type(2)));
typedef unsigned u32x2 __attribute__((ext_vector_type(2)));

#define PUB_SLEEP 2
#define MON_SLEEP 1
#define POLL_SLEEP 2
#define SKEW 8
#define OUT_AUX 2
#define LD_AUX 16
#define RSKEW 9216
#define PWAVES 3
#define PWIN 0
#define NSELF 5
constexpr int GM = 4096, GN = 16384, KS = 512, NRK = 8;
constexpr int NUNIT = (GM + GN) * KS / 1024;
constexpr unsigned REGION = NUNIT * 1024u;
constexpr unsigned RSTRIDE = REGION + RSKEW;
constexpr int GRID = 256, NCOMM = 16, NGEMM = GRID - NCOMM, NPUSH = 15, NPW = NPUSH * PWAVES;
constexpr int NPAIR = NUNIT / 2;
constexpr int NG = 160;
constexpr int ST1 = 32 - 3 * SKEW, ST2 = 32 - 2 * SKEW, ST3 = 32 - SKEW;
static_assert(SKEW == 8, "a parked strip drains in 8 K-steps");
static_assert((NSELF * NPW + 63) / 64 * 128 <= ST1 * 80, "self-formatted units must sit in the first segment");
__host__ __device__ __forceinline__ constexpr int first_of(int c) { return 2 * c - (c & 63); }
constexpr size_t WS_CTR = 0;
constexpr size_t WS_TMO = 16384;
constexpr size_t WS_CLAIM = 16384 + 256;
constexpr size_t WS_FMT = 16384 + 512;
constexpr size_t WS_FLAG = 32768;
constexpr size_t WS_PROG = 65536;
constexpr size_t WS_DATA = 1u << 20;
constexpr size_t WS_NEED = WS_DATA + 2ull * NRK * RSTRIDE;
constexpr unsigned long long DEADLINE = 100000000ull;
constexpr int LDS_BYTES = 131072;

struct Params {
    const unsigned char* x; const unsigned char* w; const float* sx; const float* sw; float* out;
    char* me;
    char* peer[2 * (NRK - 1)];
    int rank;
};

__host__ __device__ __forceinline__ constexpr int clampi(int v, int lo, int hi) { return v < lo ? lo : v > hi ? hi : v; }
__host__ __device__ __forceinline__ constexpr int cum_units(int g) {
    return 64 * clampi(g, 0, 32) + 16 * g + 15 * clampi(g - ST1, 0, 128) + 15 * clampi(g - ST2, 0, 128) + 14 * clampi(g - ST3, 0, 128);
}
static_assert(cum_units(NG) == NUNIT, "unit count");

__device__ __forceinline__ __amdgpu_buffer_rsrc_t mk_rsrc(const void* p, unsigned bytes) {
    const unsigned long long a = (unsigned long long)p;
    const unsigned long long u = ((unsigned long long)(unsigned)__builtin_amdgcn_readfirstlane((unsigned)(a >> 32)) << 32) | (unsigned)__builtin_amdgcn_readfirstlane((unsigned)a);
    return __builtin_amdgcn_make_buffer_rsrc((void*)u, 0, bytes, 0x00020000);
}
__device__ __forceinline__ unsigned region_off(unsigned call, int src) { return (unsigned)WS_DATA + ((call & 1u) * NRK + (unsigned)src) * RSTRIDE; }
constexpr int AUX_SYS = 17;
constexpr int AUX_DEV = 16;

__device__ __forceinline__ u32x4 unit_load(const Params& p, bool isx, unsigned src, int lane) {
    const auto r = mk_rsrc(isx ? p.x : p.w, isx ? GM * KS : KS * GN);
    const unsigned v = isx ? (unsigned)(4 * (lane & 15)) * KS + (unsigned)(4 * (lane >> 4)) : (unsigned)(4 * (lane >> 4)) * GN + (unsigned)(4 * (lane & 15));
    u32x4 ld;
    ld.x = __builtin_amdgcn_raw_buffer_load_b32(r, v, src, 0); ld.y = __builtin_amdgcn_raw_buffer_load_b32(r, v, src + (isx ? 2 * KS : GN), 0);
    ld.z = __builtin_amdgcn_raw_buffer_load_b32(r, v, src + (isx ? KS : 2 * GN), 0); ld.w = __builtin_amdgcn_raw_buffer_load_b32(r, v, src + (isx ? 3 * KS : 3 * GN), 0);
    return ld;
}
__device__ __forceinline__ u32x4 unit_tile(u32x4 ld, bool isx) {
    const unsigned s1 = isx ? 0x03020100u : 0x05010400u, s2 = isx ? 0x07060504u : 0x07030602u, s3 = isx ? 0x03020100u : 0x05040100u, s4 = isx ? 0x07060504u : 0x07060302u;
    const unsigned t0 = __builtin_amdgcn_perm(ld.y, ld.x, s1), t1 = __builtin_amdgcn_perm(ld.y, ld.x, s2);
    const unsigned t2 = __builtin_amdgcn_perm(ld.w, ld.z, s1), t3 = __builtin_amdgcn_perm(ld.w, ld.z, s2);
    unsigned d0 = __builtin_amdgcn_perm(t2, t0, s3), d1 = __builtin_amdgcn_perm(t2, t0, s4);
    unsigned d2 = __builtin_amdgcn_perm(t3, t1, s3), d3 = __builtin_amdgcn_perm(t3, t1, s4);
    auto r0 = __builtin_amdgcn_permlane32_swap(d0, d2, false, false); d0 = r0[0]; d2 = r0[1];
    auto r1 = __builtin_amdgcn_permlane32_swap(d1, d3, false, false); d1 = r1[0]; d3 = r1[1];
    auto r2 = __builtin_amdgcn_permlane16_swap(d0, d1, false, false); d0 = r2[0]; d1 = r2[1];
    auto r3 = __builtin_amdgcn_permlane16_swap(d2, d3, false, false); d2 = r3[0]; d3 = r3[1];
    return (u32x4){d0, d1, d2, d3};
}
__device__ __forceinline__ unsigned unit_pos(bool isx, int lane) { return (unsigned)(isx ? lane : lane ^ ((lane >> 5) * 8)) * 16u; }

__device__ __forceinline__ bool timed_out(char* ws, unsigned call, unsigned long long t0) {
    GAS unsigned* tmo = (GAS unsigned*)(ws + WS_TMO);
    if ((unsigned)__builtin_amdgcn_readfirstlane(__hip_atomic_load(tmo, __ATOMIC_RELAXED, __HIP_MEMORY_SCOPE_AGENT)) == call
        || __builtin_amdgcn_s_memrealtime() - t0 > DEADLINE) {
        __hip_atomic_store(tmo, call, __ATOMIC_RELAXED, __HIP_MEMORY_SCOPE_AGENT);
        return true;
    }
    return false;
}

__device__ __forceinline__ void pusher(const Params& p, char* myws, volatile LAS unsigned* oldest, int wid, int lane) {
    const int gw = (blockIdx.x - 1) * PWAVES + wid;
    u32x4 raw[2 * NSELF];
#pragma unroll
    for (int i = 0; i < 2 * NSELF; ++i) {
        const int n = first_of(gw + NPW * (i >> 1)) + 64 * (i & 1), g = n / 80, idx = n - g * 80;
        raw[i] = unit_load(p, idx < 64, idx < 64 ? (unsigned)idx * (64u * KS) + (unsigned)g * 16u : (unsigned)(g * 16) * GN + (unsigned)(idx < 79 ? idx - 64 : 255) * 64u, lane);
    }
    unsigned call = 0;
    if (lane == 0) call = __hip_atomic_fetch_add((GAS unsigned*)(myws + WS_CTR) + blockIdx.x * 8 + wid, 1u, __ATOMIC_RELAXED, __HIP_MEMORY_SCOPE_AGENT) + 1u;
    call = __builtin_amdgcn_readfirstlane(call);
    const unsigned dbase = region_off(call, p.rank);
    const int rot = __builtin_amdgcn_readfirstlane(gw % (NRK - 1));
    __amdgpu_buffer_rsrc_t rd[NRK - 1];
#pragma unroll
    for (int q = 0; q < NRK - 1; ++q) rd[q] = mk_rsrc(p.peer[rot + q] + dbase, REGION);
    const auto rs = mk_rsrc(myws + dbase, REGION);
    GAS unsigned* claim = (GAS unsigned*)(myws + WS_CLAIM);
    GAS unsigned* fmt = (GAS unsigned*)(myws + WS_FMT);
#define P_SEND(v, voff) do { asm volatile("" ::: "memory"); _Pragma("unroll") for (int q = 0; q < NRK - 1; ++q) __builtin_amdgcn_raw_buffer_store_b128(v, rd[q], voff, 0, AUX_SYS); \
        asm volatile("" ::: "memory"); } while (0)
#define P_PUBLISH(u) do { if (lane == 0) oldest[wid] = (unsigned)(u); } while (0)
#define P_WAIT(n) asm volatile("s_waitcnt vmcnt(%0)" :: "i"(n) : "memory")
    const unsigned cbase = (call - 1u) * (unsigned)(NPAIR - NSELF * NPW) - (NSELF + 2u) * NPW;
    int cur = gw + NPW * NSELF;
    const unsigned vl = lane * 16;
    unsigned fv = 0; u32x4 la, lb;
#pragma unroll
    for (int i = 0; i < 2 * NSELF; ++i) {
        const int n = first_of(gw + NPW * (i >> 1)) + 64 * (i & 1), idx = n % 80;
        P_SEND(unit_tile(raw[i], idx < 64), (unsigned)n * 1024u + unit_pos(idx < 64, lane));
        if (i == 2 * NSELF - 3) asm volatile("global_load_dword %0, %1, %2 sc1" : "=v"(fv) : "v"(0u), "s"(fmt) : "memory");
        if (i == 2 * NSELF - 1) lb = __builtin_amdgcn_raw_buffer_load_b128(rs, vl, first_of(cur) * 1024 + 65536, AUX_DEV);
        if (i == 2 * NSELF - 3 || i == 2 * NSELF - 1) P_WAIT(8); else asm volatile("s_waitcnt vmcnt(7)" : "+v"(fv) :: "memory");
        P_PUBLISH(min(n, first_of(gw + NPW * ((i + 1) >> 1))));
        if (i == 2 * NSELF - 2) {
            if ((int)((unsigned)__builtin_amdgcn_readfirstlane(fv) - call * NGEMM) < 0) {
                const unsigned long long t0 = __builtin_amdgcn_s_memrealtime();
                while ((int)((unsigned)__builtin_amdgcn_readfirstlane(__hip_atomic_load(fmt, __ATOMIC_RELAXED, __HIP_MEMORY_SCOPE_AGENT)) - call * NGEMM) < 0) {
                    if (timed_out(myws, call, t0)) return;
                    __builtin_amdgcn_s_sleep(1);
                }
            }
            __builtin_amdgcn_fence(__ATOMIC_ACQUIRE, "wavefront");
            la = __builtin_amdgcn_raw_buffer_load_b128(rs, vl, first_of(cur) * 1024, AUX_DEV);
        }
    }
    int nxt = cur + NPW;
    for (;;) {
        unsigned ticket;
        const unsigned vd = vl + (unsigned)first_of(cur) * 1024u;
        P_SEND(la, vd);
        la = __builtin_amdgcn_raw_buffer_load_b128(rs, vl, first_of(nxt) * 1024, AUX_DEV);
        {
            unsigned long long ex;
            asm volatile("s_mov_b64 %1, exec\n\ts_mov_b64 exec, 1\n\tv_mov_b32 %0, 1\n\tglobal_atomic_add %0, %2, %0, %3 sc0\n\ts_mov_b64 exec, %1"
                         : "=&v"(ticket), "=&s"(ex) : "v"(0u), "s"(claim) : "memory");
        }
        P_WAIT(9);
        P_PUBLISH(first_of(cur));
        P_SEND(lb, vd + 65536);
        lb = __builtin_amdgcn_raw_buffer_load_b128(rs, vl, first_of(nxt) * 1024 + 65536, AUX_DEV);
        asm volatile("s_waitcnt vmcnt(8)" : "+v"(ticket) :: "memory");
        P_PUBLISH(min(first_of(cur) + 64, first_of(nxt)));
        cur = nxt;
        nxt = (int)min((unsigned)__builtin_amdgcn_readfirstlane(ticket) - cbase, (unsigned)NPAIR);
        if (cur >= NPAIR) break;
    }
    asm volatile("s_waitcnt vmcnt(0)" ::: "memory");
    if (lane == 0) oldest[wid] = (unsigned)NUNIT;
#undef P_SEND
#undef P_PUBLISH
#undef P_WAIT
}

__device__ __forceinline__ void publisher(const Params& p, char* myws, volatile LAS unsigned* oldest, int wid, int lane) {
    unsigned call = 0;
    if (lane == 0) call = __hip_atomic_fetch_add((GAS unsigned*)(myws + WS_CTR) + blockIdx.x * 8 + wid, 1u, __ATOMIC_RELAXED, __HIP_MEMORY_SCOPE_AGENT) + 1u;
    call = __builtin_amdgcn_readfirstlane(call);
    GAS unsigned* flag = (GAS unsigned*)(p.peer[lane & 7] + WS_FLAG + (p.rank * 16 + blockIdx.x) * 128);
    const unsigned long long t0 = __builtin_amdgcn_s_memrealtime();
    unsigned last = 0;
    for (;;) {
        unsigned m = oldest[0];
#pragma unroll
        for (int i = 1; i < PWAVES; ++i) m = min(m, oldest[i]);
        m = __builtin_amdgcn_readfirstlane(m);
        if (m != last && lane < NRK - 1) __hip_atomic_store(flag, (call << 14) + m, __ATOMIC_RELAXED, __HIP_MEMORY_SCOPE_SYSTEM);
        last = m;
        if (m >= (unsigned)NUNIT || __builtin_amdgcn_s_memrealtime() - t0 > DEADLINE) break;
        __builtin_amdgcn_s_sleep(PUB_SLEEP);
    }
}

#define NMON 1
__device__ __forceinline__ void monitor(const Params& p, char* ws, unsigned call, int wid, int lane) {
    GAS const unsigned* fl[2]; bool own[2];
#pragma unroll
    for (int i = 0; i < 2; ++i) { const int f = lane + 64 * i; own[i] = (f >> 4) == p.rank || (f & 15) == 0;
        fl[i] = own[i] ? (GAS const unsigned*)(ws + WS_FMT) : (GAS const unsigned*)(ws + WS_FLAG + f * 128); }
    GAS unsigned* prog = (GAS unsigned*)(ws + WS_PROG + (size_t)(NCOMM + wid * (NGEMM / NMON) + (lane < 60 ? lane : 0)) * 128);
    const unsigned fbase = call << 14;
    const unsigned long long t0 = __builtin_amdgcn_s_memrealtime();
    int last = 0; unsigned polls = 0;
    for (;;) {
        const unsigned f0 = __hip_atomic_load(fl[0], __ATOMIC_RELAXED, __HIP_MEMORY_SCOPE_SYSTEM), f1 = __hip_atomic_load(fl[1], __ATOMIC_RELAXED, __HIP_MEMORY_SCOPE_SYSTEM);
        const int m0 = own[0] ? ((int)(f0 - call * NGEMM) < 0 ? 0 : NUNIT) : (int)(f0 - fbase), m1 = own[1] ? ((int)(f1 - call * NGEMM) < 0 ? 0 : NUNIT) : (int)(f1 - fbase);
        int m = min(m0, m1);
#pragma unroll
        for (int o = 32; o >= 1; o >>= 1) m = min(m, __shfl_xor(m, o));
        const int avail = __builtin_amdgcn_readfirstlane(m);
        if (avail > last) {
            last = avail;
            __builtin_amdgcn_fence(__ATOMIC_ACQUIRE, "");
            if (lane < 60) {
#pragma unroll
                for (int i = 0; i < 4 / NMON; ++i) __hip_atomic_store(prog + i * 60 * 32, fbase + (unsigned)avail, __ATOMIC_RELAXED, __HIP_MEMORY_SCOPE_AGENT);
            }
            if (avail >= NUNIT) break;
        } else if ((++polls & 63u) == 0u && timed_out(ws, call, t0)) break;
        __builtin_amdgcn_s_sleep(MON_SLEEP);
    }
}

__device__ __noinline__ int wait_units(char* ws, unsigned call, int need) {
    GAS const unsigned* prog = (GAS const unsigned*)(ws + WS_PROG + (size_t)blockIdx.x * 128);
    const unsigned long long t0 = __builtin_amdgcn_s_memrealtime();
    int avail; unsigned polls = 0;
    for (;;) {
        avail = (int)((unsigned)__builtin_amdgcn_readfirstlane(__hip_atomic_load(prog, __ATOMIC_RELAXED, __HIP_MEMORY_SCOPE_AGENT)) - (call << 14));
        if (avail >= need) break;
        if ((++polls & 63u) == 0u && timed_out(ws, call, t0)) { avail = -1; break; }
        __builtin_amdgcn_s_sleep(POLL_SLEEP);
    }
    __builtin_amdgcn_fence(__ATOMIC_ACQUIRE, "wavefront");
    return avail;
}

__global__ __launch_bounds__(512, 2) void fused_kernel(Params p) {
    extern __shared__ __attribute__((aligned(16))) unsigned char shm[];
    LAS unsigned char* lds = (LAS unsigned char*)shm;
    const int tid = threadIdx.x, wid = __builtin_amdgcn_readfirstlane(tid >> 6), lane = tid & 63;
    char* myws = p.me;
    constexpr bool do_push = true, do_gemm = true, do_mon = true;
    if (blockIdx.x - 1u < (unsigned)NPUSH) {
        volatile LAS unsigned* oldest = (volatile LAS unsigned*)lds;
        if (tid < 8) oldest[tid] = 0u;
        __syncthreads();
        if (wid < PWAVES && do_push) pusher(p, myws, oldest, wid, lane);
        if (wid == 3 && do_push) publisher(p, myws, oldest, wid, lane);
        return;
    }
    u32x4 raw[6];
    const int ufmt = ((blockIdx.x - NCOMM) * 8 + wid) * 6;
    if (blockIdx.x >= NCOMM && do_push) {
#pragma unroll
        for (int j = 0; j < 6; ++j) {
            const int u = min(ufmt + j, NUNIT - 1); const bool isx = u < 2048;
            raw[j] = unit_load(p, isx, isx ? (unsigned)(u >> 5) * (64u * KS) + (unsigned)(u & 31) * 16u : (unsigned)(((u - 2048) >> 8) * 16) * GN + (unsigned)(u & 255) * 64u, lane);
        }
    }
    int un;
    {
        const int u = ufmt + lane, t = (u - 2048) >> 8, s = u & 255;
        const int st = s < 75 ? 0 : s < 135 ? ST1 : s < 195 ? ST2 : ST3;
        const int r = s < 75 ? s : s < 135 ? s - 75 : s < 195 ? s - 135 : s - 195;
        const int i = r / 15, c = r - 15 * i;
        const int g = u < 2048 ? (u & 31) : s == 255 ? (t < ST3 ? t : t + 128) : st + 32 * i + ((t - st) & 31);
        const int a1 = (g >= ST1 && g < ST1 + 128) ? 15 : 0, a2 = (g >= ST2 && g < ST2 + 128) ? 15 : 0;
        const int j = (g < 32 ? 64 : 0) + c + (s < 75 ? 0 : s < 135 ? 15 : s < 195 ? 15 + a1 : 15 + a1 + a2);
        un = u >= NUNIT ? NUNIT : cum_units(g) + (u < 2048 ? (u >> 5) : j);
    }
    unsigned call = 0;
    if (lane == 0) call = __hip_atomic_fetch_add((GAS unsigned*)(myws + WS_CTR) + blockIdx.x * 8 + wid, 1u, __ATOMIC_RELAXED, __HIP_MEMORY_SCOPE_AGENT) + 1u;
    call = __builtin_amdgcn_readfirstlane(call);
    if (blockIdx.x == 0) { if (wid < NMON && do_mon) monitor(p, myws, call, wid, lane); return; }
    if (do_push) {
        const auto rloc = mk_rsrc(myws + region_off(call, p.rank), REGION);
#pragma unroll
        for (int j = 0; j < 6; ++j) {
            const bool isx = ufmt + j < 2048;
            __builtin_amdgcn_raw_buffer_store_b128(unit_tile(raw[j], isx), rloc, (unsigned)__builtin_amdgcn_readlane(un, j) * 1024u + unit_pos(isx, lane), 0, AUX_DEV);
        }
        asm volatile("s_waitcnt vmcnt(0)" ::: "memory");
        __syncthreads();
        if (tid == 0) __hip_atomic_fetch_add((GAS unsigned*)(myws + WS_FMT), 1u, __ATOMIC_RELAXED, __HIP_MEMORY_SCOPE_AGENT);
    }
    if (!do_gemm) return;
    const int lin = 30 * (blockIdx.x & 7) + (blockIdx.x >> 3) - NCOMM / 8;
    const int pm = 4 * (lin / 60) + (lin & 3), wcol = (lin % 60) >> 2;
    const int wr = wid & 3, wc = wid >> 2, fr = lane & 15, fq = lane >> 4;
    float scale = p.sx[0] * p.sw[0];
    const auto rsrc = mk_rsrc(myws + region_off(call, wid), REGION);
    const auto rout = mk_rsrc(p.out, (unsigned)GM * GN * 4u);
    GAS const unsigned* prog = (GAS const unsigned*)(myws + WS_PROG + (size_t)blockIdx.x * 128);
    const unsigned vld = lane * 16;
    const int aoff = fq * 4096 + (wr * 64 + fr) * 16;
    const int boff = 65536 + fq * 1024 + (fr & 1) * 512 + (((wc ^ (fr & 1)) * 8 + (fr >> 1)) * 16);
    const unsigned vout = (unsigned)(pm * 256 + wr * 64 + 16 * fq) * (GN * 4u) + (unsigned)(wc * 32 + 2 * fr) * 4u;
    int avail = 0;
    unsigned tabC, tabP, tabQ, tabA;
    {
        const int g = 8 * lane;
        auto start = [&](int q) { return q == 0 ? 0 : q == 1 ? ST1 : q == 2 ? ST2 : ST3; };
        auto act = [&](int q, int g) { return (q == 0 || (q == 3 && wcol == 0)) ? (unsigned)g < (unsigned)NG : (unsigned)(g - start(q)) < 128u; };
        const int qe = lane & 3;
        const int se = (qe == 3 && g <= ST3) ? 255 : (qe == 0 ? 0 : 15 + 60 * qe) + 15 * ((g - 8 - 8 * qe) >> 5) + wcol;
        const bool pe = qe == 0 ? act(0, g - 8) : qe == 1 ? act(1, g - 8) : qe == 2 ? act(2, g - 8) : act(3, g - 8);
        tabC = 128u | (act(1, g) ? 1u : 0u) | (act(2, g) ? 2u : 0u) | (act(3, g) ? 4u : 0u) | (pe ? (8u << qe) | ((unsigned)se << 8) : 0u)
             | (g < NG ? (unsigned)(cum_units(g + 1) - cum_units(g)) << 16 : 0u);
        const int i0 = g < NG ? cum_units(g) + (g < 32 ? 64 : 0) + wcol : NUNIT, a1 = act(1, g) ? 15 : 0, a2 = act(2, g) ? 15 : 0;
        const int i1 = a1 ? i0 + 15 : NUNIT, i2 = a2 ? i0 + 15 + a1 : NUNIT, i3 = act(3, g) ? i0 + 15 + a1 + a2 : NUNIT;
        tabP = (unsigned)i0 | ((unsigned)i1 << 16); tabQ = (unsigned)i2 | ((unsigned)i3 << 16);
        tabA = (g < NG ? (unsigned)(cum_units(g & 31) + 4 * pm) : (unsigned)NUNIT) | ((unsigned)(cum_units((g & 31) + 1) - cum_units(g & 31)) << 16);
    }
#define ULO(w) (((w) & 0xffffu) << 10)
#define UHI(w) (((w) >> 16) << 10)
#define GLDS(ldsoff, soff, imm) __builtin_amdgcn_raw_ptr_buffer_load_lds(rsrc, (LAS void*)(lds + (ldsoff)), 16, vld, (soff), (imm), LD_AUX)
#define STA1(b, soff, i) GLDS((b) * 32768 + wid * 4096, soff, (i) * 1024)
#define STB(b, q, soff) GLDS(65536 + ((b) * 4 + (q)) * 8192 + wid * 1024, soff, 0)
#define LD8(ptr, d) __builtin_shufflevector(*(const LAS i32x4*)(ptr), *(const LAS i32x4*)((ptr) + (d)), 0, 1, 2, 3, 4, 5, 6, 7)
#define LDA2(b, h) do { At[2 * (h)] = LD8(lds + (b) * 32768 + aoff + (h) * 512, 16384); At[2 * (h) + 1] = LD8(lds + (b) * 32768 + aoff + (h) * 512 + 256, 16384); } while (0)
#define LDB1(b, q, n) Bt[(q) & 1][n] = LD8(lds + ((b) * 4 + (q)) * 8192 + (n) * 256 + boff, 4096)
#define LDB(b, q) do { LDB1(b, q, 0); LDB1(b, q, 1); } while (0)
#define MMA(q) do { _Pragma("unroll") for (int n = 0; n < 2; ++n) _Pragma("unroll") for (int mm = 0; mm < 4; ++mm) { const int m = n ? 3 - mm : mm;     \
        acc[q][m][n] = __builtin_amdgcn_mfma_scale_f32_16x16x128_f8f6f4(At[m], Bt[(q) & 1][n], acc[q][m][n], 0, 1, 0, 0, 0, 0); SCHED; } } while (0)
#define PIECE(v, p) ((f32x2){(v)[(p) >> 2][0][(p) & 3], (v)[(p) >> 2][1][(p) & 3]})
#define POFF(p) ((unsigned)(4 * ((p) & 3) + ((p) >> 2)) * (GN * 4u))
#define STO(v, so) __builtin_amdgcn_raw_buffer_store_b64(__builtin_bit_cast(u32x2, (v)), rout, vout, (so), OUT_AUX)
#define DRIP(p) STO(pk[p], so + POFF(p))
#define PARK(q) do { _Pragma("unroll") for (int i = 0; i < 16; ++i) { float _x, _y; \
            asm("v_mul_f32 %0, %1, %2" : "=v"(_x) : "v"(acc[q][i >> 2][0][i & 3]), "v"(scale)); asm("v_mul_f32 %0, %1, %2" : "=v"(_y) : "v"(acc[q][i >> 2][1][i & 3]), "v"(scale)); pk[i] = (f32x2){_x, _y}; } \
        _Pragma("unroll") for (int m = 0; m < 4; ++m) _Pragma("unroll") for (int n = 0; n < 2; ++n) acc[q][m][n] = (f32x4){0.f, 0.f, 0.f, 0.f}; } while (0)
#define WAIT_V(n) asm volatile("s_waitcnt vmcnt(" #n ")" ::: "memory")
#define WAIT_L0 __builtin_amdgcn_s_waitcnt(0xC07F)
#define BAR __builtin_amdgcn_s_barrier()
#define SCHED __builtin_amdgcn_sched_barrier(0)
#define PRIO(n) __builtin_amdgcn_s_setprio(n)
#define NEED() do { need += (int)dn; if (need > avail) { avail = __builtin_amdgcn_readfirstlane(wait_units(myws, call, need)); \
        if (avail < 0) { avail = 0x7fffffff; scale = __builtin_nanf(""); } } } while (0)
#define STAMP() do {} while (0)
    STAMP();
    f32x4 acc[4][4][2]; f32x2 pk[16];
#pragma unroll
    for (int i = 0; i < 16; ++i) { pk[i] = (f32x2){0.f, 0.f}; acc[i >> 2][i & 3][0] = (f32x4){0.f, 0.f, 0.f, 0.f}; acc[i >> 2][i & 3][1] = (f32x4){0.f, 0.f, 0.f, 0.f}; }
    i32x8 At[4], Bt[2][2];
    constexpr unsigned SO_OFF = 0x40000000u;
    unsigned so = SO_OFF;
    unsigned ctl = __builtin_amdgcn_readlane(tabC, 0), nctl, nq;
    unsigned ua, u0, u1, u2, u3, da, dn = ctl >> 16, d23 = UHI(ctl);
    int need = 0;
    { const unsigned aw = __builtin_amdgcn_readlane(tabA, 0), pw = __builtin_amdgcn_readlane(tabP, 0), qw = __builtin_amdgcn_readlane(tabQ, 0);
      ua = ULO(aw); da = UHI(aw); u0 = ULO(pw); u1 = UHI(pw); u2 = ULO(qw); u3 = UHI(qw); }
    NEED();
    STA1(0, ua, 0); STA1(0, ua, 1); STA1(0, ua, 2); STA1(0, ua, 3); STB(0, 0, u0); STB(0, 1, u1); SCHED; STB(0, 2, u2); STB(0, 3, u3); SCHED; DRIP(0); DRIP(0); SCHED;
    ua += da; u0 += d23; u1 += d23; u2 += d23; u3 += d23;
    NEED();
    STA1(1, ua, 0); STA1(1, ua, 1); STA1(1, ua, 2); STA1(1, ua, 3); STB(1, 0, u0); STB(1, 1, u1);
    ua += da; u0 += d23; u1 += d23;
    unsigned pv = call << 14;
    WAIT_V(10); BAR;
    if (wc == 1) BAR;
    STAMP();
#define KSTEP(k) do { \
        if ((k) == 7) { u2 = ULO(nq); u3 = UHI(nq); d23 = UHI(nctl); } \
        if ((k) & 1) asm volatile("global_load_dword %0, %1, %2 sc1" : "=v"(pv) : "v"(0u), "s"(prog) : "memory"); \
        LDB((k) & 1, 0); SCHED; STB(1 - ((k) & 1), 2, u2); SCHED; LDA2((k) & 1, 0); SCHED; STB(1 - ((k) & 1), 3, u3); SCHED; LDA2((k) & 1, 1); SCHED; \
        DRIP((2 * (k) + 14) & 15); SCHED; LDB((k) & 1, 1); SCHED; DRIP((2 * (k) + 15) & 15); SCHED; \
        u2 += d23; u3 += d23; \
        if ((k) == 0) { \
            if (ctl & 8u) PARK(0); \
            if (ctl & 16u) PARK(1); \
            if (ctl & 32u) PARK(2); \
            if (ctl & 64u) PARK(3); \
            so = (ctl & 0x78u) ? (ctl & 0xff00u) : SO_OFF; \
        } \
        if ((k) == 5) { nctl = __builtin_amdgcn_readlane(tabC, body + 1); nq = __builtin_amdgcn_readlane(tabQ, body + 1); } \
        if ((k) == 6) { const unsigned aw = __builtin_amdgcn_readlane(tabA, body + 1), pw = __builtin_amdgcn_readlane(tabP, body + 1); \
                        ua = ULO(aw); da = UHI(aw); u0 = ULO(pw); u1 = UHI(pw); dn = nctl >> 16; } \
        SCHED; \
        if ((k) & 1) WAIT_V(13); else { asm volatile("s_waitcnt vmcnt(12)" : "+v"(pv) :: "memory");     \
                                       avail = max(avail, (int)((unsigned)__builtin_amdgcn_readfirstlane(pv) - (call << 14))); } \
        WAIT_L0; BAR; PRIO(1); if (ctl & 128u) MMA(0); if (ctl & 1u) MMA(1); PRIO(0); BAR; SCHED; \
        NEED(); \
        LDB1((k) & 1, 2, 0); SCHED; STA1((k) & 1, ua, 0); SCHED; LDB1((k) & 1, 2, 1); SCHED; STA1((k) & 1, ua, 1); SCHED; LDB1((k) & 1, 3, 0); SCHED; STA1((k) & 1, ua, 2); SCHED; \
        LDB1((k) & 1, 3, 1); SCHED; STA1((k) & 1, ua, 3); STB((k) & 1, 0, u0); STB((k) & 1, 1, u1); SCHED; \
        ua += da; u0 += dn << 10; u1 += dn << 10; \
        if ((k) & 1) WAIT_V(11); else WAIT_V(10); \
        WAIT_L0; BAR; PRIO(1); if (ctl & 2u) MMA(2); if (ctl & 4u) MMA(3); PRIO(0); BAR; SCHED; } while (0)
    for (int body = 0; body < NG / 8; ++body) {
        KSTEP(0); KSTEP(1); KSTEP(2); KSTEP(3); KSTEP(4); KSTEP(5); KSTEP(6); KSTEP(7);
        ctl = nctl;
        STAMP();
    }
    DRIP(14); DRIP(15); SCHED;
    f32x2 head[16];
    if (wcol == 0) {
#pragma unroll
        for (int i = 0; i < 16; ++i) head[i] = __builtin_bit_cast(f32x2, __builtin_amdgcn_raw_buffer_load_b64(rout, vout, 255u * 256u + POFF(i), AUX_DEV));
    }
#pragma unroll
    for (int i = 0; i < 16; ++i) STO(PIECE(acc[0], i) * scale, (60u + wcol) * 256u + POFF(i));
    if (wcol == 0) {
#pragma unroll
        for (int i = 0; i < 16; ++i) STO(PIECE(acc[3], i) * scale + head[i], 255u * 256u + POFF(i));
    }
    asm volatile("s_waitcnt vmcnt(0)" : "+v"(pv) :: "memory");
    STAMP();
    if (wc == 0) BAR;
}
}

extern "C" void kernel_launch(int rank, int nranks, ncclComm_t comm,
                              hipStream_t stream,
                              void* const* d_in, const int* in_sizes, int n_in,
                              void* d_out, int out_size,
                              const int* in_dtypes, int out_dtype,
                              void* const* d_out_all,
                              void* const* d_ws_all, size_t ws_bytes) {
    using namespace gk;
    if (nranks != NRK || (unsigned)rank >= (unsigned)NRK || n_in != 4 || ws_bytes < WS_NEED ||
        in_sizes[0] != GM * KS || in_sizes[1] != KS * GN || out_size != GM * GN) {
        fprintf(stderr, "kernel_launch: unexpected problem shape\n"); return;
    }
    static bool attr_set = false;
    if (!attr_set) {
        HIP_CHECK(hipFuncSetAttribute((const void*)fused_kernel, hipFuncAttributeMaxDynamicSharedMemorySize, LDS_BYTES));
        attr_set = true;
    }
    Params p{};
    p.x = (const unsigned char*)d_in[0]; p.w = (const unsigned char*)d_in[1];
    p.sx = (const float*)d_in[2]; p.sw = (const float*)d_in[3];
    p.out = (float*)d_out; p.rank = rank;
    p.me = (char*)d_ws_all[rank];
    for (int q = 0; q < 2 * (NRK - 1); ++q) p.peer[q] = (char*)d_ws_all[(rank + 1 + q % (NRK - 1)) % NRK];
    hipLaunchKernelGGL(fused_kernel, dim3(GRID), dim3(512), LDS_BYTES, stream, p);
    HIP_CHECK(hipGetLastError());
}
```

```cpp
#include <hip/hip_runtime.h>
#include <rccl/rccl.h>
#include <cstdio>
#include <cstdint>

#define HIP_CHECK(expr) do { hipError_t _e = (expr); if (_e != hipSuccess) { \
    fprintf(stderr, "HIP error %s at %s:%d\n", hipGetErrorString(_e), __FILE__, __LINE__); \
    return; } } while (0)

namespace gk {
#define LAS __attribute__((address_space(3)))
#define GAS __attribute__((address_space(1)))
typedef int i32x4 __attribute__((ext_vector_type(4)));
typedef int i32x8 __attribute__((ext_vector_type(8)));
typedef unsigned u32x4 __attribute__((ext_vector_type(4)));
typedef float f32x4 __attribute__((ext_vector_type(4)));
typedef float f32x2 __attribute__((ext_vector_type(2)));
typedef unsigned u32x2 __attribute__((ext_vector_type(2)));

#define PUB_SLEEP 4
#define PUB_LAZY 120
#define PUB_LO (unsigned)(cum_units(34) - 700)
#define PUB_HI (unsigned)(cum_units(34) + 900)
#define MON_SLEEP 4
#define POLL_SLEEP 2
#define NPOLL 2
#define NPROG 64
#define OUT_AUX 2
#define LD_AUX 16
#define RSKEW 9216
#define NSELF 5
#define SAHEAD 4
#define PWIN 7
#define SWIN 10
#define SK 8
#define OWNS 2
constexpr int NG = 128 + 3 * SK;
constexpr int PWAVES = 3;
constexpr int GM = 4096, GN = 16384, KS = 512, NRK = 8;
constexpr int NSRC = (GM + GN) * KS / 1024;
constexpr int NREG = 256 - NRK * OWNS, NX = 64 + NREG - 240;
constexpr int NOPD = (GM + NREG * 64) * KS / 1024;
constexpr int NOWN = OWNS * 32;
constexpr int NOUT = GM * OWNS * 256 / 1024;
constexpr int NSTREAM = NOPD + 1;
constexpr unsigned REGION = (NOPD + NOWN) * 1024u;
constexpr unsigned RSTRIDE = REGION + RSKEW;
constexpr int GRID = 256, NCOMM = 16, NGEMM = GRID - NCOMM, NPUSH = 15, NPW = NPUSH * PWAVES;
constexpr int NPAIR = NOPD / 2;
static_assert(NOPD % 128 == 0 && NSTREAM < 16384, "");
static_assert((NSELF * NPW + 63) / 64 * 128 <= SK * (2 * NX + 45), "self-formatted units must sit in the first two schedule segments");
__host__ __device__ __forceinline__ constexpr int first_of(int c) { return 2 * c - (c & 63); }
constexpr size_t WS_CTR = 0;
constexpr size_t WS_TMO = 16384;
constexpr size_t WS_CLAIM = 16384 + 256;
constexpr size_t WS_FMT = 20480;
constexpr size_t WS_FMT2 = 20480 + 1024;
constexpr size_t WS_SHR = 20480 + 2048;
constexpr size_t WS_FLAG = 32768;
constexpr size_t WS_PROG = 65536;
constexpr size_t WS_STAGE = 1u << 20;
constexpr size_t WS_DATA = 3u << 20;
constexpr size_t WS_NEED = WS_DATA + 2ull * NRK * RSTRIDE;
constexpr unsigned long long DEADLINE = 100000000ull;
constexpr int LDS_SEEN = 131072 + 3 * 8192, LDS_BYTES = LDS_SEEN + 16;
constexpr unsigned OUT_BYTES = (unsigned)GM * GN * 4u;

struct Params {
    const unsigned char* x; const unsigned char* w; const float* sx; const float* sw; float* out;
    char* me;
    char* ws[NRK];
    char* peer[2 * (NRK - 1)];
    char* pout[2 * (NRK - 1)];
    int rank;
};

__host__ __device__ __forceinline__ constexpr int clampi(int v, int lo, int hi) { return v < lo ? lo : v > hi ? hi : v; }
__host__ __device__ __forceinline__ constexpr int cum_units(int g) {
    return NX * clampi(g, 0, 32) + 15 * (clampi(g, 0, 128) + clampi(g - SK, 0, 128) + clampi(g - 2 * SK, 0, 128) + clampi(g - 3 * SK, 0, 128));
}
static_assert(cum_units(NG) == NOPD, "unit count");
__device__ __forceinline__ int w_unit(int q, int c, int g) {
    return cum_units(g) + (g < 32 ? NX : 0) + c + 15 * (g < 128 ? q : q - 1 - (g - 128) / SK);
}

__device__ __forceinline__ char* uni(const void* p) {
    const unsigned long long a = (unsigned long long)p;
    return (char*)(((unsigned long long)(unsigned)__builtin_amdgcn_readfirstlane((unsigned)(a >> 32)) << 32) | (unsigned)__builtin_amdgcn_readfirstlane((unsigned)a));
}
__device__ __forceinline__ __amdgpu_buffer_rsrc_t mk_rsrc(const void* p, unsigned bytes) { return __builtin_amdgcn_make_buffer_rsrc(uni(p), 0, bytes, 0x00020000); }
__device__ __forceinline__ unsigned region_off(unsigned call, int src) { return (unsigned)WS_DATA + ((call & 1u) * NRK + (unsigned)src) * RSTRIDE; }
constexpr int AUX_SYS = 17;
constexpr int AUX_DEV = 16;

struct XW { const unsigned char* x; const unsigned char* w; };
__device__ __forceinline__ u32x4 unit_load(const XW& p, bool isx, unsigned src, int lane) {
    const auto r = __builtin_amdgcn_make_buffer_rsrc((void*)(isx ? p.x : p.w), 0, isx ? GM * KS : KS * GN, 0x00020000);
    const unsigned v = isx ? (unsigned)(4 * (lane & 15)) * KS + (unsigned)(4 * (lane >> 4)) : (unsigned)(4 * (lane >> 4)) * GN + (unsigned)(4 * (lane & 15));
    u32x4 ld;
    ld.x = __builtin_amdgcn_raw_buffer_load_b32(r, v, src, 0); ld.y = __builtin_amdgcn_raw_buffer_load_b32(r, v, src + (isx ? 2 * KS : GN), 0);
    ld.z = __builtin_amdgcn_raw_buffer_load_b32(r, v, src + (isx ? KS : 2 * GN), 0); ld.w = __builtin_amdgcn_raw_buffer_load_b32(r, v, src + (isx ? 3 * KS : 3 * GN), 0);
    return ld;
}
__device__ __forceinline__ u32x4 unit_tile(u32x4 ld, bool isx) {
    const unsigned s1 = isx ? 0x03020100u : 0x05010400u, s2 = isx ? 0x07060504u : 0x07030602u, s3 = isx ? 0x03020100u : 0x05040100u, s4 = isx ? 0x07060504u : 0x07060302u;
    const unsigned t0 = __builtin_amdgcn_perm(ld.y, ld.x, s1), t1 = __builtin_amdgcn_perm(ld.y, ld.x, s2);
    const unsigned t2 = __builtin_amdgcn_perm(ld.w, ld.z, s1), t3 = __builtin_amdgcn_perm(ld.w, ld.z, s2);
    unsigned d0 = __builtin_amdgcn_perm(t2, t0, s3), d1 = __builtin_amdgcn_perm(t2, t0, s4);
    unsigned d2 = __builtin_amdgcn_perm(t3, t1, s3), d3 = __builtin_amdgcn_perm(t3, t1, s4);
    auto r0 = __builtin_amdgcn_permlane32_swap(d0, d2, false, false); d0 = r0[0]; d2 = r0[1];
    auto r1 = __builtin_amdgcn_permlane32_swap(d1, d3, false, false); d1 = r1[0]; d3 = r1[1];
    auto r2 = __builtin_amdgcn_permlane16_swap(d0, d1, false, false); d0 = r2[0]; d1 = r2[1];
    auto r3 = __builtin_amdgcn_permlane16_swap(d2, d3, false, false); d2 = r3[0]; d3 = r3[1];
    return (u32x4){d0, d1, d2, d3};
}
__device__ __forceinline__ unsigned unit_pos(bool isx, int lane) { return (unsigned)(isx ? lane : lane ^ ((lane >> 5) * 8)) * 16u; }

__device__ __forceinline__ unsigned draw_async(GAS unsigned* ctr, unsigned inc = 1u) {
    unsigned t; unsigned long long ex;
    asm volatile("s_mov_b64 %1, exec\n\ts_mov_b64 exec, 1\n\tv_mov_b32 %0, %4\n\ts_nop 1\n\tglobal_atomic_add %0, %2, %0, %3 sc0\n\ts_mov_b64 exec, %1"
                 : "=&v"(t), "=&s"(ex) : "v"(0u), "s"(ctr), "s"(inc) : "memory");
    return t;
}
__device__ __forceinline__ unsigned draw_call(const Params& p, int wid, int lane) {
    unsigned call = 0;
    if (lane == 0) call = __hip_atomic_fetch_add((GAS unsigned*)(p.me + WS_CTR) + blockIdx.x * 8 + wid, 1u, __ATOMIC_RELAXED, __HIP_MEMORY_SCOPE_AGENT) + 1u;
    call = __builtin_amdgcn_readfirstlane(call);
    return call;
}
__device__ __forceinline__ bool timed_out(char* ws, unsigned call, unsigned long long t0) {
    GAS unsigned* tmo = (GAS unsigned*)(ws + WS_TMO);
    if ((unsigned)__builtin_amdgcn_readfirstlane(__hip_atomic_load(tmo, __ATOMIC_RELAXED, __HIP_MEMORY_SCOPE_AGENT)) == call
        || __builtin_amdgcn_s_memrealtime() - t0 > DEADLINE) {
        __hip_atomic_store(tmo, call, __ATOMIC_RELAXED, __HIP_MEMORY_SCOPE_AGENT);
        return true;
    }
    return false;
}
__device__ __forceinline__ bool wait_count(char* ws, size_t off, unsigned want, unsigned call, int lane) {
    const unsigned long long t0 = __builtin_amdgcn_s_memrealtime();
    unsigned polls = 0;
    while (__any((int)(__hip_atomic_load((GAS unsigned*)(ws + off + (lane & 7) * 128), __ATOMIC_RELAXED, __HIP_MEMORY_SCOPE_AGENT) - want) < 0)) {
        if ((++polls & 15u) == 0u && timed_out(ws, call, t0)) return false;
        __builtin_amdgcn_s_sleep(2);
    }
    __builtin_amdgcn_fence(__ATOMIC_ACQUIRE, "wavefront");
    return true;
}

__device__ __forceinline__ void pusher(const Params& p, const XW& xw, volatile LAS unsigned* oldest, int wid, int lane) {
    const int gw = (blockIdx.x - 1) * PWAVES + wid;
    unsigned cv = draw_async((GAS unsigned*)uni(p.me + WS_CTR + (blockIdx.x * 8 + wid) * 4));
    u32x4 raw[2 * NSELF];
    int vn; unsigned vs;
    {
        vn = first_of(gw + NPW * ((lane & 15) >> 1)) + 64 * (lane & 1);
        const bool s0 = vn < (NX + 15) * SK;
        const int r = s0 ? vn : vn - (NX + 15) * SK, q = (r * (s0 ? 1048576 / (NX + 15) + 1 : 1048576 / (NX + 30) + 1)) >> 20;
        const int g = s0 ? q : SK + q, idx = r - q * (s0 ? NX + 15 : NX + 30);
        vs = idx < 64 ? ((unsigned)idx * (64u * KS) + (unsigned)g * 16u) | 0x80000000u
                      : (unsigned)(g * 16) * GN + (unsigned)(idx < NX ? idx + 176 : idx < NX + 15 ? idx - NX : idx - NX + 45) * 64u;
    }
    auto raw_load = [&](int i) { const unsigned sv = __builtin_amdgcn_readlane(vs, i); raw[i] = unit_load(xw, (int)sv < 0, sv & 0x7fffffffu, lane); };
#pragma unroll
    for (int i = 0; i < SAHEAD; ++i) raw_load(i);
    __builtin_amdgcn_s_barrier();
    asm volatile("s_waitcnt vmcnt(%1)" : "+v"(cv) : "i"(4 * SAHEAD) : "memory");
    unsigned call = __builtin_amdgcn_readfirstlane(cv) + 1u;
    const unsigned dbase = region_off(call, p.rank);
    const int rot = __builtin_amdgcn_readfirstlane(gw % (NRK - 1));
    __amdgpu_buffer_rsrc_t rd[NRK - 1];
#pragma unroll
    for (int q = 0; q < NRK - 1; ++q) rd[q] = mk_rsrc(p.peer[rot + q] + dbase, NOPD * 1024u);
    const auto rs = mk_rsrc(p.me + dbase, NOPD * 1024u);
    GAS unsigned* claim = (GAS unsigned*)uni(p.me + WS_CLAIM);
    GAS unsigned* fmt = (GAS unsigned*)uni(p.me + WS_FMT);
#define P_SEND(v, voff) do { asm volatile("" ::: "memory"); _Pragma("unroll") for (int q = 0; q < NRK - 1; ++q) __builtin_amdgcn_raw_buffer_store_b128(v, rd[q], voff, 0, AUX_SYS); \
        asm volatile("" ::: "memory"); } while (0)
#define P_PUBLISH(u) do { if (lane == 0) oldest[wid] = (unsigned)(u); } while (0)
#define P_WAIT(n) asm volatile("s_waitcnt vmcnt(%0)" :: "i"(n) : "memory")
    const unsigned cbase = (call - 1u) * (unsigned)(NPAIR - NSELF * NPW) - (NSELF + 3u) * NPW;
    int cur = gw + NPW * NSELF;
    const unsigned vl = lane * 16;
    unsigned fv = 0; u32x4 la[2], lb[2];
#define P_LOAD(c, h) __builtin_amdgcn_raw_buffer_load_b128(rs, vl, first_of(c) * 1024 + (h) * 65536, AUX_DEV)
#pragma unroll
    for (int i = 0; i < 2 * NSELF; ++i) {
        const int n = __builtin_amdgcn_readlane(vn, i); const bool isx = (int)__builtin_amdgcn_readlane(vs, i) < 0;
        P_SEND(unit_tile(raw[i], isx), (unsigned)n * 1024u + unit_pos(isx, lane));
        if (i + SAHEAD < 2 * NSELF) raw_load(i + SAHEAD);
        if (i == 2 * NSELF - 3) asm volatile("s_nop 4\n\tglobal_load_dword %0, %1, %2 sc1" : "=v"(fv) : "v"((lane & 7) * 128), "s"(fmt) : "memory");
        if (i >= 2 * NSELF - 2) {
            asm volatile("s_waitcnt vmcnt(7)" : "+v"(fv) :: "memory");
            P_PUBLISH(min(n, first_of(gw + NPW * ((i + 1) >> 1))));
        }
        if (i == 2 * NSELF - 2) {
            if (__any((int)(fv - call * (NGEMM / 8)) < 0) && !wait_count(p.me, WS_FMT, call * (NGEMM / 8), call, lane)) return;
            la[0] = P_LOAD(cur, 0); lb[0] = P_LOAD(cur, 1); la[1] = P_LOAD(cur + NPW, 0); lb[1] = P_LOAD(cur + NPW, 1);
        }
    }
    asm volatile("" : "+v"(la[0]), "+v"(lb[0]), "+v"(la[1]), "+v"(lb[1]));
    int nxt = cur + NPW;
    unsigned ticket = cbase + cur + 2 * NPW;
    int pub = min(first_of(cur - NPW) + 64, first_of(cur));
#define P_TRIP(h) { \
        const int nx2 = (int)min((unsigned)__builtin_amdgcn_readfirstlane(ticket) - cbase, (unsigned)NPAIR); \
        ticket = draw_async(claim, cur < NPAIR ? 1u : 0u);     \
        const unsigned vd = vl + (unsigned)first_of(cur) * 1024u; \
        P_SEND(la[h], vd); \
        la[h] = P_LOAD(nx2, 0); \
        P_WAIT(9 + PWIN); \
        P_PUBLISH(PWIN ? pub : first_of(cur)); \
        P_SEND(lb[h], vd + 65536); \
        lb[h] = P_LOAD(nx2, 1); \
        pub = min(first_of(cur) + 64, first_of(nxt)); \
        asm volatile("s_waitcnt vmcnt(%1)" : "+v"(ticket) : "i"(8 + PWIN) : "memory"); \
        P_PUBLISH(PWIN ? first_of(cur) : pub); \
        cur = nxt; nxt = nx2; }
    do { P_TRIP(0) P_TRIP(1) } while (cur < NPAIR);
#undef P_TRIP
#undef P_LOAD
    if (lane == 0) oldest[8 + wid] = 1u;
    asm volatile("s_waitcnt vmcnt(0)" ::: "memory");
    P_PUBLISH(NSTREAM);
#undef P_SEND
#undef P_PUBLISH
#undef P_WAIT
}

constexpr int OPW = (NOUT + NPW - 1) / NPW;
__device__ __forceinline__ void out_sender(const Params& p, volatile LAS unsigned* oldest, int wid, int lane) {
    const int gw = (blockIdx.x - 1) * PWAVES + wid - 4;
    const unsigned call = draw_call(p, wid, lane);
    const unsigned long long t0 = __builtin_amdgcn_s_memrealtime();
    while (oldest[wid - 4] < 6400u + (unsigned)gw * 48u) {
        if (__builtin_amdgcn_s_memrealtime() - t0 > DEADLINE) return;
        __builtin_amdgcn_s_sleep(32);
    }
    if (!wait_count(p.me, WS_SHR, call * (16u * OWNS), call, lane)) return;
    const auto rg = mk_rsrc(p.me + WS_STAGE, NOUT * 1024u);
    const unsigned vo = (unsigned)(gw * (4 / OWNS) * OPW + lane / (16 * OWNS)) * (GN * 4u) + (unsigned)(NREG + OWNS * p.rank) * 256u + (unsigned)(lane % (16 * OWNS)) * 16u;
    u32x4 v[OPW];
#pragma unroll
    for (int i = 0; i < OPW; ++i) v[i] = __builtin_amdgcn_raw_buffer_load_b128(rg, lane * 16, (unsigned)(gw * OPW + i) << 10, AUX_DEV);
#pragma unroll
    for (int i = 0; i < OPW; ++i) asm volatile("" : "+v"(v[i]));
    const int rot = __builtin_amdgcn_readfirstlane(gw % (NRK - 1));
    __amdgpu_buffer_rsrc_t rd[NRK - 1];
#pragma unroll
    for (int q = 0; q < NRK - 1; ++q) rd[q] = mk_rsrc(p.pout[rot + q], OUT_BYTES);
    while (oldest[4 + wid] == 0u) {
        if (__builtin_amdgcn_s_memrealtime() - t0 > DEADLINE) return;
        __builtin_amdgcn_s_sleep(8);
    }
#pragma unroll
    for (int i = 0; i < OPW; ++i) {
#pragma unroll
        for (int q = 0; q < NRK - 1; ++q) __builtin_amdgcn_raw_buffer_store_b128(v[i], rd[q], vo + (unsigned)i * (4u / OWNS * GN * 4u), 0, AUX_SYS);
        asm volatile("s_waitcnt vmcnt(%0)" :: "i"(SWIN) : "memory");
    }
    asm volatile("s_waitcnt vmcnt(0)" ::: "memory");
    if (lane == 0) oldest[wid - 1] = (unsigned)NSTREAM;
}

__device__ __forceinline__ void publisher(const Params& p, volatile LAS unsigned* oldest, int wid, int lane) {
    const unsigned call = draw_call(p, wid, lane);
    GAS unsigned* flag = (GAS unsigned*)(p.peer[lane & 7] + WS_FLAG + (p.rank * 16 + blockIdx.x) * 128);
    if (!wait_count(p.me, WS_FMT2, call * (NGEMM / 8), call, lane)) return;
    const unsigned long long t0 = __builtin_amdgcn_s_memrealtime();
    unsigned last = 0;
    for (;;) {
        unsigned m = oldest[0];
#pragma unroll
        for (int i = 1; i < 2 * PWAVES; ++i) m = min(m, oldest[i]);
        m = __builtin_amdgcn_readfirstlane(m);
        if (m != last && lane < NRK - 1) __hip_atomic_store(flag, (call << 14) + m, __ATOMIC_RELAXED, __HIP_MEMORY_SCOPE_SYSTEM);
        last = m;
        if (m >= (unsigned)NSTREAM || __builtin_amdgcn_s_memrealtime() - t0 > DEADLINE) break;
        if ((m >= PUB_LO && m < PUB_HI) || m >= (unsigned)NOPD) __builtin_amdgcn_s_sleep(PUB_SLEEP); else __builtin_amdgcn_s_sleep(PUB_LAZY);
    }
}

__device__ __forceinline__ void poller(const Params& p, LAS int* seen, unsigned call, int lane) {
    char* ws = p.me;
    GAS const unsigned* fl[2]; bool own[2];
#pragma unroll
    for (int i = 0; i < 2; ++i) { const int f = lane + 64 * i; own[i] = (f >> 4) == p.rank || (f & 15) == 0;
        fl[i] = own[i] ? (GAS const unsigned*)(ws + WS_FMT + (f & 7) * 128) : (GAS const unsigned*)(ws + WS_FLAG + f * 128); }
    const unsigned fbase = call << 14;
    const unsigned long long t0 = __builtin_amdgcn_s_memrealtime();
    int last = 0;
    for (;;) {
        const unsigned f0 = __hip_atomic_load(fl[0], __ATOMIC_RELAXED, __HIP_MEMORY_SCOPE_SYSTEM), f1 = __hip_atomic_load(fl[1], __ATOMIC_RELAXED, __HIP_MEMORY_SCOPE_SYSTEM);
        const int m0 = own[0] ? ((int)(f0 - call * (NGEMM / 8)) < 0 ? 0 : NSTREAM) : (int)(f0 - fbase), m1 = own[1] ? ((int)(f1 - call * (NGEMM / 8)) < 0 ? 0 : NSTREAM) : (int)(f1 - fbase);
        int m = min(m0, m1);
#pragma unroll
        for (int o = 32; o >= 1; o >>= 1) m = min(m, __shfl_xor(m, o));
        const int avail = __builtin_amdgcn_readfirstlane(m);
        if (avail > last) {
            last = avail;
            if (avail >= NSTREAM) __builtin_amdgcn_fence(__ATOMIC_ACQUIRE, "");
            if (lane == 0) __hip_atomic_fetch_max(seen, avail, __ATOMIC_RELAXED, __HIP_MEMORY_SCOPE_WORKGROUP);
            if (avail >= NSTREAM) break;
        }
        if (__builtin_amdgcn_s_memrealtime() - t0 > DEADLINE) break;
        __builtin_amdgcn_s_sleep(MON_SLEEP);
    }
}
__device__ __forceinline__ void monitor(const Params& p, volatile LAS int* seen, unsigned call, int lane) {
    GAS unsigned* prog = (GAS unsigned*)(p.me + WS_PROG + (size_t)lane * 128);
    const unsigned long long t0 = __builtin_amdgcn_s_memrealtime();
    int last = 0; unsigned polls = 0;
    for (;;) {
        const int avail = __builtin_amdgcn_readfirstlane(*seen);
        if (avail > last) {
            if (last < NOPD && lane < NPROG) __hip_atomic_store(prog, (call << 14) + (unsigned)avail, __ATOMIC_RELAXED, __HIP_MEMORY_SCOPE_AGENT);
            last = avail;
            if (avail >= NSTREAM) break;
        } else if ((++polls & 1023u) == 0u && timed_out(p.me, call, t0)) {
            p.out[lane * 256] = __builtin_nanf("");
            break;
        }
        __builtin_amdgcn_s_sleep(1);
    }
}

__device__ __noinline__ int wait_units(char* ws, unsigned call, int need) {
    GAS const unsigned* prog = (GAS const unsigned*)(ws + WS_PROG + (size_t)(blockIdx.x % NPROG) * 128);
    const unsigned long long t0 = __builtin_amdgcn_s_memrealtime();
    int avail; unsigned polls = 0;
    for (;;) {
        avail = (int)((unsigned)__builtin_amdgcn_readfirstlane(__hip_atomic_load(prog, __ATOMIC_RELAXED, __HIP_MEMORY_SCOPE_AGENT)) - (call << 14));
        if (avail >= need) break;
        if ((++polls & 63u) == 0u && timed_out(ws, call, t0)) { avail = -1; break; }
        __builtin_amdgcn_s_sleep(POLL_SLEEP);
    }
    __builtin_amdgcn_fence(__ATOMIC_ACQUIRE, "wavefront");
    return avail;
}

__global__ __launch_bounds__(512, 2) void fused_kernel(Params p) {
    extern __shared__ __attribute__((aligned(16))) unsigned char shm[];
    LAS unsigned char* lds = (LAS unsigned char*)shm;
    const int tid = threadIdx.x, wid = __builtin_amdgcn_readfirstlane(tid >> 6), lane = tid & 63;
    char* myws = p.me;
    XW xw{p.x, p.w};
    asm volatile("" : "+s"(xw.x), "+s"(xw.w), "+s"(myws));
    if (__builtin_expect(blockIdx.x - 1u < (unsigned)NPUSH, 1)) {
        volatile LAS unsigned* oldest = (volatile LAS unsigned*)lds;
        if (__builtin_expect(wid < PWAVES, 1)) { pusher(p, xw, oldest, wid, lane); return; }
        if (wid == 3 && lane < 12) oldest[lane] = (lane >= PWAVES && lane < 2 * PWAVES) ? (unsigned)NOPD : 0u;
        __syncthreads();
        if (wid == 3) publisher(p, oldest, wid, lane);
        if (wid >= 4 && wid < 4 + PWAVES) out_sender(p, oldest, wid, lane);
        return;
    }
    u32x4 raw[6];
    const int vw = (blockIdx.x - NCOMM) * 8 + wid;
    auto fmt_unit = [&](int j) { return j == 0 ? vw : j < 5 ? 2047 + vw * 4 + j : vw < 128 ? 1920 + vw : 9600 + vw; };
    if (blockIdx.x >= NCOMM) {
#pragma unroll
        for (int j = 0; j < 6; ++j) {
            const int u = min(fmt_unit(j), NSRC - 1); const bool isx = u < 2048;
            raw[j] = unit_load(xw, isx, isx ? (unsigned)(u >> 5) * (64u * KS) + (unsigned)(u & 31) * 16u : (unsigned)(((u - 2048) >> 8) * 16) * GN + (unsigned)(u & 255) * 64u, lane);
        }
    }
    int un;
    {
        const int u = fmt_unit(lane), t = (u - 2048) >> 8, s = u & 255;
        const int q = s / 60, r = s - 60 * q, i = r / 15, c = r - 15 * i;
        const int g = SK * q + 32 * i + ((t - SK * q) & 31);
        un = u >= NSRC ? NSRC : u < 2048 ? cum_units(u & 31) + (u >> 5) : s >= NREG ? NOPD + (s % OWNS) * 32 + t : s >= 240 ? cum_units(t) + s - 176 : w_unit(q, c, g);
    }
    const unsigned call = draw_call(p, wid, lane);
    if (blockIdx.x == 0) {
        if (tid == 0) *(LAS int*)lds = 0;
        __syncthreads();
        if (wid == 0) monitor(p, (volatile LAS int*)lds, call, lane);
        if (wid >= 1 && wid <= NPOLL) poller(p, (LAS int*)lds, call, lane);
        return;
    }
    if (tid == 0) *(LAS int*)(lds + LDS_SEEN) = 0;
    {
#pragma unroll
        for (int j = 0; j < 6; ++j) raw[j] = unit_tile(raw[j], fmt_unit(j) < 2048);
#pragma unroll
        for (int pass = 0; pass < 2; ++pass) {
#pragma unroll
            for (int j = pass; j < 6; ++j) {
                const int u = fmt_unit(j); const bool isx = u < 2048;
                const int owner = (!isx && (u & 255) >= NREG) ? ((u & 255) - NREG) / OWNS : p.rank;
                if ((owner == p.rank) == (pass == 0)) {
                    const auto rloc = mk_rsrc((pass ? p.ws[owner & 7] : myws) + region_off(call, p.rank), REGION);
                    __builtin_amdgcn_raw_buffer_store_b128(raw[j], rloc, (unsigned)__builtin_amdgcn_readlane(un, j) * 1024u + unit_pos(isx, lane), 0, AUX_SYS);
                }
            }
            asm volatile("s_waitcnt vmcnt(0)" ::: "memory");
            __syncthreads();
            if (tid == 0) __hip_atomic_fetch_add((GAS unsigned*)(myws + (pass ? WS_FMT2 : WS_FMT) + (blockIdx.x & 7) * 128), 1u, __ATOMIC_RELAXED, __HIP_MEMORY_SCOPE_AGENT);
        }
    }
    const int lin = 30 * (blockIdx.x & 7) + (blockIdx.x >> 3) - NCOMM / 8;
    const int pm = 4 * (lin / 60) + (lin & 3), wcol = 14 - ((lin % 60) >> 2);
    const int wr = wid & 3, wc = wid >> 2, fr = lane & 15, fq = lane >> 4;
    float scale = p.sx[0] * p.sw[0];
    const auto rsrc = mk_rsrc(myws + region_off(call, wid), REGION);
    const auto rout = mk_rsrc(p.out, OUT_BYTES);
    GAS const unsigned* prog = (GAS const unsigned*)uni(myws + WS_PROG + (size_t)(blockIdx.x % NPROG) * 128);
    const unsigned vld = lane * 16;
    const int aoff = fq * 4096 + (wr * 64 + fr) * 16;
    const int boff = 65536 + fq * 1024 + (fr & 1) * 512 + (((wc ^ (fr & 1)) * 8 + (fr >> 1)) * 16);
    const unsigned vout = (unsigned)(pm * 256 + wr * 64 + 16 * fq) * (GN * 4u) + (unsigned)(wc * 32 + 2 * fr) * 4u;
    int avail = 0;
    unsigned tabP, tabQ, tabA;
    {
        const int g = SK * lane;
        auto pos = [&](int q) { return (unsigned)((unsigned)(g - SK * q) < 128u ? w_unit(q, wcol, g) : NSRC); };
        tabP = pos(0) | (pos(2) << 16); tabQ = pos(1) | (pos(3) << 16);
        tabA = g < NG ? (unsigned)(cum_units(g & 31) + 4 * pm) | ((unsigned)(cum_units((g & 31) + 1) - cum_units(g & 31)) << 16) | ((unsigned)(cum_units(g + 1) - cum_units(g)) << 24) : (unsigned)NSRC;
    }
#define ULO(w) (((w) & 0xffffu) << 10)
#define UHI(w) (((w) >> 16) << 10)
#define GLDS(ldsoff, soff, imm) __builtin_amdgcn_raw_ptr_buffer_load_lds(rsrc, (LAS void*)(lds + (ldsoff)), 16, vld, (soff), (imm), LD_AUX)
#define STA1(b, soff, i) GLDS((b) * 32768 + wid * 4096, soff, (i) * 1024)
#define STB(b, q, soff) GLDS(65536 + ((b) * 4 + (q)) * 8192 + wid * 1024, soff, 0)
#define LD8(ptr, d) __builtin_shufflevector(*(const LAS i32x4*)(ptr), *(const LAS i32x4*)((ptr) + (d)), 0, 1, 2, 3, 4, 5, 6, 7)
#define LDA2(b, h) do { At[2 * (h)] = LD8(lds + (b) * 32768 + aoff + (h) * 512, 16384); At[2 * (h) + 1] = LD8(lds + (b) * 32768 + aoff + (h) * 512 + 256, 16384); } while (0)
#define LDB1(b, q, n) Bt[(q) & 1][n] = LD8(lds + ((b) * 4 + (q)) * 8192 + (n) * 256 + boff, 4096)
#define LDB(b, q) do { LDB1(b, q, 0); LDB1(b, q, 1); } while (0)
#define MMA(c, B, z) do { _Pragma("unroll") for (int i = 0; i < 8; ++i) { const int n = i >> 2, m = n ? 7 - i : i; \
        if (z) asm volatile("v_mfma_f32_16x16x128_f8f6f4 %0, %1, %2, 0 blgp:1" : "=&v"(c[m][n]) : "v"(At[m]), "v"(B[n])); \
        else asm volatile("v_mfma_f32_16x16x128_f8f6f4 %0, %1, %2, %0 blgp:1" : "+v"(c[m][n]) : "v"(At[m]), "v"(B[n])); } } while (0)
#define PROW(p) ((unsigned)(4 * ((p) & 3) + ((p) >> 2)))
#define DRIP(p) __builtin_amdgcn_raw_buffer_store_b64(__builtin_bit_cast(u32x2, pk[p]), rout, vout, so + PROW(p) * (GN * 4u), OUT_AUX)
#define PARK(c) do { _Pragma("unroll") for (int i = 0; i < 16; ++i) { float _x, _y; \
            asm volatile("v_mul_f32 %0, %1, %2" : "=v"(_x) : "v"(c[i >> 2][0][i & 3]), "v"(scale)); asm volatile("v_mul_f32 %0, %1, %2" : "=v"(_y) : "v"(c[i >> 2][1][i & 3]), "v"(scale)); pk[i] = (f32x2){_x, _y}; } } while (0)
#define WAIT_VN(n) asm volatile("s_waitcnt vmcnt(%0)" :: "i"(n) : "memory")
#define WAIT_V(n) asm volatile("s_waitcnt vmcnt(" #n ")" ::: "memory")
#define WAIT_L0 __builtin_amdgcn_s_waitcnt(0xC07F)
#define BAR __builtin_amdgcn_s_barrier()
#define SCHED __builtin_amdgcn_sched_barrier(0)
#define PRIO(n) __builtin_amdgcn_s_setprio(n)
#define NEED() do { need += (int)dn; if (__builtin_expect(need > avail, 0)) { avail = __builtin_amdgcn_readfirstlane(*(volatile LAS int*)(lds + LDS_SEEN)); \
        if (need > avail) { asm volatile("s_waitcnt vmcnt(0)" : "+v"(pv) :: "memory");     \
            avail = __builtin_amdgcn_readfirstlane(wait_units(myws, call, need)); \
            *(volatile LAS int*)(lds + LDS_SEEN) = avail; \
            if (avail < 0) { avail = 0x7fffffff; scale = __builtin_nanf(""); } } } } while (0)
    f32x4 acc[4][4][2], own[4][2]; f32x2 pk[16];
    i32x8 At[4], Bt[2][2];
    unsigned so = 0, nq;
    unsigned ua, u0, u1, u2, u3, u4 = (unsigned)(NOPD + wcol * 32) << 10, da, dn, d23;
    int need = 0, blk = 0;
    unsigned pv = call << 14;
    { const unsigned aw = __builtin_amdgcn_readlane(tabA, 0), pw = __builtin_amdgcn_readlane(tabP, 0), qw = __builtin_amdgcn_readlane(tabQ, 0);
      ua = ULO(aw); da = ((aw >> 16) & 255u) << 10; dn = aw >> 24; d23 = dn << 10; u0 = ULO(pw); u1 = UHI(pw); u2 = ULO(qw); u3 = UHI(qw); }
    const bool five = wcol < OWNS + NREG - 240;
    const unsigned c4 = five ? (unsigned)(64 + wcol - OWNS - 4 * pm) << 10 : (unsigned)NSRC << 10;
#define STW(g) do { GLDS(131072 + ((g) % 3) * 8192 + wid * 1024, wcol < OWNS ? u4 : ua + c4, 0); u4 += 1024; } while (0)
    {
        unsigned long long c0; int clen;
        asm volatile("s_getpc_b64 %0\ngk_code_lo:\n\ts_mov_b32 %1, gk_code_hi-gk_code_lo" : "=s"(c0), "=s"(clen));
        const auto rc = __builtin_amdgcn_make_buffer_rsrc((void*)c0, 0, max(clen, 0), 0x00020000);
        const unsigned a = __builtin_amdgcn_raw_buffer_load_b32(rc, vld * 8, wid * 16384, 0) ^ __builtin_amdgcn_raw_buffer_load_b32(rc, vld * 8, wid * 16384 + 8192, 0);
        asm volatile("" :: "v"(a));
    }
    NEED();
    if (blockIdx.x - (NCOMM + 1u) < (unsigned)NPUSH && wid < NRK - 1) {
        const unsigned row0 = (blockIdx.x - (NCOMM + 1u)) * (4u / OWNS * PWAVES * OPW);
        const auto r1 = mk_rsrc(p.pout[wid], OUT_BYTES), r2 = mk_rsrc(p.peer[wid] + region_off(call, p.rank), REGION);
        const unsigned a = __builtin_amdgcn_raw_buffer_load_b32(r1, lane < 10 ? min(row0 + lane * 32u, row0 + 4u / OWNS * PWAVES * OPW - 1u) * (GN * 4u) : OUT_BYTES, 0, AUX_SYS)
                         ^ __builtin_amdgcn_raw_buffer_load_b32(r2, lane < 6 ? min((unsigned)lane << 21, REGION - 4u) : REGION, 0, AUX_SYS);
        asm volatile("" :: "v"(a));
    }
    STA1(0, ua, 0); STA1(0, ua, 1); STA1(0, ua, 2); STA1(0, ua, 3); STB(0, 0, u0); STB(0, 1, u1); STW(0); SCHED; STB(0, 2, u2); STB(0, 3, u3); SCHED;
    ua += da; u0 += d23; u1 += d23; u2 += d23; u3 += d23;
    NEED();
    STA1(1, ua, 0); STA1(1, ua, 1); STA1(1, ua, 2); STA1(1, ua, 3); STB(1, 0, u0); STB(1, 1, u1); STW(1);
    ua += da; u0 += d23; u1 += d23;
    WAIT_V(9); BAR;
    if (wc == 1) BAR;
#define SET_OF(l) ((l) == 1 ? 2 : (l) == 2 ? 1 : (l))
#define ACT(sec, k, r) ((sec) == 1 || ((sec) == 0 ? (k) >= SK * SET_OF(r) : (k) < SK * SET_OF(r)))
#define FRESH(sec, k, r) ((k) == SK * SET_OF(r))
#define NST(sec, k) (((sec) > 0 && (k) >= 1 && (k) <= 4 * SK) || (SK == 8 && (k) == 0 && (sec) > 0) ? 16 / SK : 0)
#define NSTP(sec, k) ((k) > 0 ? NST(sec, (k) - 1) : 0)
#define NB(sec, k) (((sec) == 0 && (k) < 30) ? 7 : 6)
#define NBP(sec, k) (((sec) == 0 && (k) <= 30) ? 7 : 6)
#define PC0(k) ((((k) + SK - 1) % SK) * (16 / SK))
#define DRIPN(sec, k, j) do { if ((j) < NST(sec, k)) { DRIP((PC0(k) + (j)) & 15); SCHED; } } while (0)
#define KSTEP(sec, k) do { \
        if ((k) % SK == SK - 1) { u2 = ULO(nq); u3 = UHI(nq); d23 = dn << 10; } \
        if ((k) & 1) asm volatile("s_nop 4\n\tglobal_load_dword %0, %1, %2 sc1" : "=v"(pv) : "v"(0u), "s"(prog) : "memory"); \
        if (ACT(sec, k, 0)) LDB((k) & 1, 0); SCHED; STB(1 - ((k) & 1), 2, u2); SCHED; LDA2((k) & 1, 0); SCHED; STB(1 - ((k) & 1), 3, u3); SCHED; DRIPN(sec, k, 2); LDA2((k) & 1, 1); SCHED; \
        DRIPN(sec, k, 0); if (ACT(sec, k, 1)) LDB1((k) & 1, 1, 0); SCHED; DRIPN(sec, k, 3); if (ACT(sec, k, 1)) LDB1((k) & 1, 1, 1); SCHED; DRIPN(sec, k, 1); \
        u2 += d23; u3 += d23; \
        if ((sec) > 0 && (k) % SK == 0 && (k) < 4 * SK) {     \
            PARK(acc[SET_OF(((k) / SK) & 3)]); SCHED; so = (unsigned)(60 * ((k) / SK) + strip0) * 256u; } \
        if ((sec) == 1 && (k) == 4 * SK - 1) strip0 += 15; \
          \
        if ((sec) == 1 && (k) == 6 && __builtin_expect(strip0 < OWNS, 0) && lane == 0) __hip_atomic_fetch_add((GAS unsigned*)(myws + WS_SHR + wid * 128), 1u, __ATOMIC_RELAXED, __HIP_MEMORY_SCOPE_AGENT); \
        if ((k) % SK == SK - 2) { ++blk; const unsigned aw = __builtin_amdgcn_readlane(tabA, blk), pw = __builtin_amdgcn_readlane(tabP, blk); nq = __builtin_amdgcn_readlane(tabQ, blk); \
                        ua = ULO(aw); da = ((aw >> 16) & 255u) << 10; dn = aw >> 24; u0 = ULO(pw); u1 = UHI(pw); } \
        SCHED; \
        if ((k) & 1) WAIT_VN(NSTP(sec, k) + NBP(sec, k) + 3 + NST(sec, k)); \
        else { asm volatile("s_waitcnt vmcnt(%1)" : "+v"(pv) : "i"(NSTP(sec, k) + NBP(sec, k) + 2 + NST(sec, k)) : "memory");     \
               avail = max(avail, (int)((unsigned)__builtin_amdgcn_readfirstlane(pv) - (call << 14))); } \
        WAIT_L0; BAR; PRIO(1); if (ACT(sec, k, 0)) MMA(acc[0], Bt[0], FRESH(sec, k, 0)); if (ACT(sec, k, 1)) MMA(acc[1], Bt[1], FRESH(sec, k, 1)); PRIO(0); BAR; SCHED; \
        NEED(); \
        if (ACT(sec, k, 2)) LDB1((k) & 1, 2, 0); SCHED; STA1((k) & 1, ua, 0); SCHED; if (ACT(sec, k, 2)) LDB1((k) & 1, 2, 1); SCHED; STA1((k) & 1, ua, 1); SCHED; \
        if (ACT(sec, k, 3)) LDB1((k) & 1, 3, 0); SCHED; STA1((k) & 1, ua, 2); SCHED; if (ACT(sec, k, 3)) LDB1((k) & 1, 3, 1); SCHED; \
        STA1((k) & 1, ua, 3); STB((k) & 1, 0, u0); STB((k) & 1, 1, u1); if (NB(sec, k) == 7) STW((k) + 2); SCHED; \
        ua += da; u0 += dn << 10; u1 += dn << 10; \
        WAIT_VN(((k) & 1) + 2 + NST(sec, k) + NB(sec, k)); \
        WAIT_L0; BAR; PRIO(1); if (ACT(sec, k, 2)) MMA(acc[2], Bt[0], FRESH(sec, k, 2)); if (ACT(sec, k, 3)) MMA(acc[3], Bt[1], FRESH(sec, k, 3)); \
        if ((sec) == 0 && five) {                      \
            Bt[0][0] = LD8(lds + 65536 + ((k) % 3) * 8192 + boff, 4096); Bt[0][1] = LD8(lds + 65536 + ((k) % 3) * 8192 + 256 + boff, 4096); \
            _Pragma("unroll") for (int i = 0; i < 8; ++i) own[i & 3][i >> 2] = __builtin_amdgcn_mfma_scale_f32_16x16x128_f8f6f4(At[i & 3], Bt[0][i >> 2], own[i & 3][i >> 2], 0, 1, 0, 0, 0, 0); \
            WAIT_L0; } \
        PRIO(0); BAR; SCHED; } while (0)
#define K4(sec, k) do { KSTEP(sec, k); KSTEP(sec, (k) + 1); KSTEP(sec, (k) + 2); KSTEP(sec, (k) + 3); } while (0)
#define K12(sec, k) do { K4(sec, k); K4(sec, (k) + 4); K4(sec, (k) + 8); } while (0)
    int strip0 = wcol;
#pragma unroll
    for (int i = 0; i < 8; ++i) { own[i >> 1][i & 1] = (f32x4){0.f, 0.f, 0.f, 0.f}; asm volatile("" : "+v"(own[i >> 1][i & 1])); }
    K12(0, 0); K12(0, 12); K4(0, 24); K4(0, 28);
    asm volatile("s_nop 15" ::: "memory"); SCHED;
    PARK(own);
    so = 0x40000000u;
    if (five) {
        const auto rg = mk_rsrc(myws + WS_STAGE, wcol < OWNS ? NOUT * 1024u : 0u); const unsigned vg = (vout >> 16) * (OWNS * 256u) + (vout & 255u) + wcol * 256u;
        const unsigned s5 = (unsigned)(wcol < OWNS ? NREG + OWNS * p.rank + wcol : 240 + wcol - OWNS) * 256u;
#pragma unroll
        for (int i = 0; i < 16; ++i) {
            __builtin_amdgcn_raw_buffer_store_b64(__builtin_bit_cast(u32x2, pk[i]), rg, vg, PROW(i) * (OWNS * 256u), AUX_DEV);
            __builtin_amdgcn_raw_buffer_store_b64(__builtin_bit_cast(u32x2, pk[i]), rout, vout, s5 + PROW(i) * (GN * 4u), OUT_AUX);
        }
    }
    SCHED;
    do { K12(1, 0); K12(1, 12); K4(1, 24); K4(1, 28); } while (strip0 < 45);
    K12(2, 0);
    if (SK == 8) K12(2, 12);
    asm volatile("gk_code_hi:");
#pragma unroll
    for (int j = 0; j < 16 / SK; ++j) DRIP(16 - 16 / SK + j);
    asm volatile("s_nop 15" ::: "memory"); SCHED;
    PARK(acc[3]);
    so = (unsigned)(180 + strip0) * 256u;
#pragma unroll
    for (int i = 0; i < 16; ++i) DRIP(i);
    asm volatile("s_waitcnt vmcnt(0)" : "+v"(pv) :: "memory");
    if (wc == 0) BAR;
}
}

extern "C" void kernel_launch(int rank, int nranks, ncclComm_t comm,
                              hipStream_t stream,
                              void* const* d_in, const int* in_sizes, int n_in,
                              void* d_out, int out_size,
                              const int* in_dtypes, int out_dtype,
                              void* const* d_out_all,
                              void* const* d_ws_all, size_t ws_bytes) {
    using namespace gk;
    if (nranks != NRK || (unsigned)rank >= (unsigned)NRK || n_in != 4 || ws_bytes < WS_NEED ||
        in_sizes[0] != GM * KS || in_sizes[1] != KS * GN || out_size != GM * GN) {
        fprintf(stderr, "kernel_launch: unexpected problem shape\n"); return;
    }
    static bool attr_set = false;
    if (!attr_set) {
        HIP_CHECK(hipFuncSetAttribute((const void*)fused_kernel, hipFuncAttributeMaxDynamicSharedMemorySize, LDS_BYTES));
        attr_set = true;
    }
    Params p{};
    p.x = (const unsigned char*)d_in[0]; p.w = (const unsigned char*)d_in[1];
    p.sx = (const float*)d_in[2]; p.sw = (const float*)d_in[3];
    p.out = (float*)d_out; p.rank = rank;
    p.me = (char*)d_ws_all[rank];
    for (int q = 0; q < NRK; ++q) p.ws[q] = (char*)d_ws_all[q];
    for (int q = 0; q < 2 * (NRK - 1); ++q) {
        p.peer[q] = (char*)d_ws_all[(rank + 1 + q % (NRK - 1)) % NRK];
        p.pout[q] = (char*)d_out_all[(rank + 1 + q % (NRK - 1)) % NRK];
    }
    hipLaunchKernelGGL(fused_kernel, dim3(GRID), dim3(512), LDS_BYTES, stream, p);
    HIP_CHECK(hipGetLastError());
}
```
